```python
import numpy as np

kernel_name = "gpu_gemm_ar_m4096_k4096_n32768_f32_relu"


ABSMAX_RTOL = 0.001

COMPUTE_DTYPE = "float32"
EPILOGUE = "relu"
IN_QUANT = None
OUT_QUANT = None
IN_DTYPES = {"x": "float32", "w_mat": "float32"}
WORLD_SIZE = 8

def setup_inputs():

    x = np.random.randn(4096, 4096).astype(np.float32)
    w_mat = (np.random.randn(4096, 32768) / np.sqrt(4096)).astype(np.float32)
    return {"x": x, "w_mat": w_mat}

def _epilogue(y):
    return np.maximum(y, 0.0).astype(np.float32)

def reference(x, w_mat):
    y = (x @ w_mat).astype(np.float32)
    return _epilogue(y)
```

```cpp
#include <hip/hip_runtime.h>
#include <rccl/rccl.h>
#include <cstdio>
#include <cstdint>

namespace {

constexpr int NR   = 8;
constexpr int GM   = 4096;
constexpr int GK   = 4096;
constexpr int GN   = 32768;
constexpr int KS   = GK / NR;
constexpr int NL   = GN / NR;

constexpr int BM = 128, BN = 128, BK = 32;
constexpr int APAD = 4;
constexpr int AST  = BK + APAD;
constexpr int LDS_A = BM * AST;
constexpr int LDS_B = BK * BN;
constexpr int LDS_FLOATS = 2 * (LDS_A + LDS_B);
constexpr int CST = BN + 4;
static_assert(BM * CST <= LDS_FLOATS, "C staging must fit");

constexpr int TILES_M = GM / BM;
constexpr int TILES_N = NL / BN;
constexpr int NTILES  = TILES_M * TILES_N;

constexpr size_t WS_CTRL   = 0;
constexpr size_t WS_X      = 1u << 20;
constexpr size_t WS_W      = WS_X + (size_t)GM * GK * 4;
constexpr size_t WS_END    = WS_W + (size_t)GK * NL * 4;

constexpr int LINE = 64;
constexpr int C_TICKET0 = 0, C_TICKET1 = 1, C_TICKET2 = 2, C_TMO = 3;
constexpr int C_IN   = 8;
constexpr int C_DONE = 16;

constexpr unsigned long long DEADLINE_TICKS = 100000000ull;

constexpr int G0 = 1024;
constexpr int G1 = NTILES;

typedef float f32x16 __attribute__((ext_vector_type(16)));

struct Params {
    const float* x;
    const float* w;
    float* out[NR];
    char*  ws[NR];
    int rank;
};

__device__ __forceinline__ unsigned* ctrl(char* ws, int idx) {
    return reinterpret_cast<unsigned*>(ws + WS_CTRL) + idx * LINE;
}

__device__ __forceinline__ bool wait_ge(unsigned* flag, unsigned target, unsigned* tmo, unsigned call) {
    unsigned long long t0 = __builtin_amdgcn_s_memrealtime();
    while (__hip_atomic_load(flag, __ATOMIC_RELAXED, __HIP_MEMORY_SCOPE_SYSTEM) < target) {
        if (__hip_atomic_load(tmo, __ATOMIC_RELAXED, __HIP_MEMORY_SCOPE_AGENT) == call ||
            __builtin_amdgcn_s_memrealtime() - t0 > DEADLINE_TICKS) {
            __hip_atomic_store(tmo, call, __ATOMIC_RELAXED, __HIP_MEMORY_SCOPE_AGENT);
            return false;
        }
        __builtin_amdgcn_s_sleep(2);
    }
    return true;
}

__device__ __forceinline__ void publish_all(const Params& p, int idx) {
    asm volatile("s_waitcnt vmcnt(0)" ::: "memory");
    __syncthreads();
    if (threadIdx.x == 0) {
        __builtin_amdgcn_fence(__ATOMIC_RELEASE, "");
        asm volatile("s_waitcnt vmcnt(0)" ::: "memory");
        #pragma unroll
        for (int q = 1; q < NR; ++q) {
            int d = (p.rank + q) & (NR - 1);
            __hip_atomic_fetch_add(ctrl(p.ws[d], idx + p.rank), 1u, __ATOMIC_RELAXED, __HIP_MEMORY_SCOPE_SYSTEM);
        }
    }
}

__global__ void __launch_bounds__(256, 2) fused_kernel(Params p, int phase) {
    extern __shared__ __attribute__((aligned(16))) float lds[];
    __shared__ unsigned s_call;
    __shared__ int s_ok;
    char* myws = p.ws[p.rank];
    const int tid = threadIdx.x;

    if (phase == 0) {
        const int dst = (blockIdx.x + p.rank) & (NR - 1);
        const int wg  = blockIdx.x >> 3, nwg = gridDim.x >> 3;
        char* dws = p.ws[dst];
        float* xs = reinterpret_cast<float*>(dws + WS_X);
        float* wsb = reinterpret_cast<float*>(dws + WS_W);
        for (int u = wg; u < GM / 2; u += nwg) {
            int row = 2 * u + (tid >> 7), c4 = tid & 127;
            float4 v = reinterpret_cast<const float4*>(p.x + (size_t)row * KS)[c4];
            reinterpret_cast<float4*>(xs + (size_t)row * GK + p.rank * KS)[c4] = v;
        }
        for (int u = wg; u < KS * 4; u += nwg) {
            int k = u >> 2, part = u & 3;
            float4 v = reinterpret_cast<const float4*>(p.w + (size_t)k * GN + (size_t)dst * NL + part * 1024)[tid];
            reinterpret_cast<float4*>(wsb + (size_t)(p.rank * KS + k) * NL + part * 1024)[tid] = v;
        }
        asm volatile("s_waitcnt vmcnt(0)" ::: "memory");
        __syncthreads();
        if (tid == 0 && dst != p.rank) {
            __builtin_amdgcn_fence(__ATOMIC_RELEASE, "");
            asm volatile("s_waitcnt vmcnt(0)" ::: "memory");
            __hip_atomic_fetch_add(ctrl(dws, C_IN + p.rank), 1u, __ATOMIC_RELAXED, __HIP_MEMORY_SCOPE_SYSTEM);
        }
        return;
    }

    if (phase == 2) {
        if (tid == 0) {
            unsigned t = __hip_atomic_fetch_add(ctrl(myws, C_TICKET2), 1u, __ATOMIC_RELAXED, __HIP_MEMORY_SCOPE_AGENT);
            unsigned call = t / gridDim.x + 1;
            bool ok = true;
            for (int q = 1; q < NR; ++q) {
                int s = (p.rank + q) & (NR - 1);
                ok = wait_ge(ctrl(myws, C_DONE + s), (unsigned)G1 * call, ctrl(myws, C_TMO), call) && ok;
            }
            __builtin_amdgcn_fence(__ATOMIC_ACQUIRE, "");
            if (!ok || __hip_atomic_load(ctrl(myws, C_TMO), __ATOMIC_RELAXED, __HIP_MEMORY_SCOPE_AGENT) == call)
                p.out[p.rank][(size_t)p.rank * NL] = __builtin_nanf("");
        }
        return;
    }

    if (tid == 0) {
        unsigned t = __hip_atomic_fetch_add(ctrl(myws, C_TICKET1), 1u, __ATOMIC_RELAXED, __HIP_MEMORY_SCOPE_AGENT);
        unsigned call = t / gridDim.x + 1;
        bool ok = true;
        for (int q = 1; q < NR; ++q) {
            int s = (p.rank + q) & (NR - 1);
            ok = wait_ge(ctrl(myws, C_IN + s), (unsigned)(G0 / NR) * call, ctrl(myws, C_TMO), call) && ok;
        }
        __builtin_amdgcn_fence(__ATOMIC_ACQUIRE, "");
        asm volatile("s_waitcnt vmcnt(0)" ::: "memory");
        ok = ok && (__hip_atomic_load(ctrl(myws, C_TMO), __ATOMIC_RELAXED, __HIP_MEMORY_SCOPE_AGENT) != call);
        s_call = call;
        s_ok = ok ? 1 : 0;
    }
    __syncthreads();
    const bool ok = s_ok != 0;

    const int b = blockIdx.x;
    const int xcd = b & 7, idx = b >> 3;
    const int tm = xcd * 4 + (idx & 3), tn = idx >> 2;
    const int m0 = tm * BM, n0 = tn * BN;

    const float* A = reinterpret_cast<const float*>(myws + WS_X);
    const float* B = reinterpret_cast<const float*>(myws + WS_W);

    const int wave = tid >> 6, lane = tid & 63;
    const int wm = wave >> 1, wn = wave & 1;
    const int l31 = lane & 31, h = lane >> 5;

    float* As = lds;
    float* Bs = lds + 2 * LDS_A;

    f32x16 acc[2][2];
    #pragma unroll
    for (int i = 0; i < 2; ++i)
        #pragma unroll
        for (int j = 0; j < 2; ++j)
            #pragma unroll
            for (int r = 0; r < 16; ++r) acc[i][j][r] = 0.f;

    float4 ra[4], rb[4];
    const int a_row = tid >> 3, a_c4 = tid & 7;
    const int b_row = tid >> 5, b_c4 = tid & 31;

    auto load_g = [&](int kt) {
        const int k0 = kt * BK;
        #pragma unroll
        for (int i = 0; i < 4; ++i)
            ra[i] = *reinterpret_cast<const float4*>(A + (size_t)(m0 + a_row + 32 * i) * GK + k0 + a_c4 * 4);
        #pragma unroll
        for (int i = 0; i < 4; ++i)
            rb[i] = *reinterpret_cast<const float4*>(B + (size_t)(k0 + b_row + 8 * i) * NL + n0 + b_c4 * 4);
    };
    auto store_l = [&](int buf) {
        float* a = As + buf * LDS_A;
        float* bb = Bs + buf * LDS_B;
        #pragma unroll
        for (int i = 0; i < 4; ++i)
            *reinterpret_cast<float4*>(a + (a_row + 32 * i) * AST + a_c4 * 4) = ra[i];
        #pragma unroll
        for (int i = 0; i < 4; ++i)
            *reinterpret_cast<float4*>(bb + (b_row + 8 * i) * BN + b_c4 * 4) = rb[i];
    };

    constexpr int NT = GK / BK;
    if (ok) {
        load_g(0);
        store_l(0);
        __syncthreads();
        for (int kt = 0; kt < NT; ++kt) {
            const int buf = kt & 1;
            if (kt + 1 < NT) load_g(kt + 1);
            const float* a = As + buf * LDS_A;
            const float* bb = Bs + buf * LDS_B;
            #pragma unroll
            for (int kg = 0; kg < BK / 8; ++kg) {
                float4 af[2];
                float bf[2][4];
                #pragma unroll
                for (int bi = 0; bi < 2; ++bi)
                    af[bi] = *reinterpret_cast<const float4*>(a + (wm * 64 + bi * 32 + l31) * AST + kg * 8 + 4 * h);
                #pragma unroll
                for (int bj = 0; bj < 2; ++bj)
                    #pragma unroll
                    for (int s = 0; s < 4; ++s)
                        bf[bj][s] = bb[(kg * 8 + 4 * h + s) * BN + wn * 64 + bj * 32 + l31];
                #pragma unroll
                for (int s = 0; s < 4; ++s) {
                    #pragma unroll
                    for (int bi = 0; bi < 2; ++bi) {
                        float av = s == 0 ? af[bi].x : s == 1 ? af[bi].y : s == 2 ? af[bi].z : af[bi].w;
                        #pragma unroll
                        for (int bj = 0; bj < 2; ++bj)
                            acc[bi][bj] = __builtin_amdgcn_mfma_f32_32x32x2f32(av, bf[bj][s], acc[bi][bj], 0, 0, 0);
                    }
                }
            }
            if (kt + 1 < NT) store_l(buf ^ 1);
            __syncthreads();
        }
    }

    float* Cs = lds;
    #pragma unroll
    for (int bi = 0; bi < 2; ++bi)
        #pragma unroll
        for (int bj = 0; bj < 2; ++bj)
            #pragma unroll
            for (int r = 0; r < 16; ++r) {
                int row = wm * 64 + bi * 32 + (r & 3) + 8 * (r >> 2) + 4 * h;
                int col = wn * 64 + bj * 32 + l31;
                float v = acc[bi][bj][r];
                Cs[row * CST + col] = ok ? fmaxf(v, 0.f) : __builtin_nanf("");
            }
    __syncthreads();
    {
        const int c4 = tid & 31, r0 = tid >> 5;
        #pragma unroll 4
        for (int i = 0; i < BM / 8; ++i) {
            int row = r0 + 8 * i;
            float4 v = *reinterpret_cast<const float4*>(Cs + row * CST + c4 * 4);
            size_t off = (size_t)(m0 + row) * GN + (size_t)p.rank * NL + n0 + c4 * 4;
            #pragma unroll
            for (int q = 0; q < NR; ++q) {
                int d = (p.rank + q) & (NR - 1);
                *reinterpret_cast<float4*>(p.out[d] + off) = v;
            }
        }
    }
    publish_all(p, C_DONE);
}

}

extern "C" void kernel_launch(int rank, int nranks, ncclComm_t comm,
                              hipStream_t stream,
                              void* const* d_in, const int* in_sizes, int n_in,
                              void* d_out, int out_size,
                              const int* in_dtypes, int out_dtype,
                              void* const* d_out_all,
                              void* const* d_ws_all, size_t ws_bytes) {
    (void)comm; (void)in_sizes; (void)n_in; (void)out_size; (void)in_dtypes; (void)out_dtype; (void)d_out;
    if (nranks != NR || ws_bytes < WS_END) {
        fprintf(stderr, "kernel_launch: unsupported configuration (nranks=%d ws=%zu)\n", nranks, ws_bytes);
        return;
    }
    Params p;
    p.x = static_cast<const float*>(d_in[0]);
    p.w = static_cast<const float*>(d_in[1]);
    for (int i = 0; i < NR; ++i) {
        p.out[i] = static_cast<float*>(d_out_all[i]);
        p.ws[i]  = static_cast<char*>(d_ws_all[i]);
    }
    p.rank = rank;
    static bool attr_done = false;
    if (!attr_done) {
        hipFuncSetAttribute(reinterpret_cast<const void*>(fused_kernel),
                            hipFuncAttributeMaxDynamicSharedMemorySize, LDS_FLOATS * 4);
        attr_done = true;
    }
    fused_kernel<<<G0, 256, 0, stream>>>(p, 0);
    fused_kernel<<<G1, 256, LDS_FLOATS * 4, stream>>>(p, 1);
    fused_kernel<<<1, 64, 0, stream>>>(p, 2);
}
```

```cpp
#include <hip/hip_runtime.h>
#include <rccl/rccl.h>
#include <cstdio>
#include <cstdint>

namespace {

constexpr int NR   = 8;
constexpr int GM   = 4096;
constexpr int GK   = 4096;
constexpr int GN   = 32768;
constexpr int KS   = GK / NR;
constexpr int NL   = GN / NR;

constexpr int BM = 128, BN = 128, BK = 16;
constexpr int LDS_A = BM * BK;
constexpr int LDS_B = BK * BN;
constexpr int NBUF = 4;

constexpr int CST = BN + 4;
constexpr int CROWS = 64;
constexpr int LDS_FLOATS = (NBUF * (LDS_A + LDS_B) > CROWS * CST) ? NBUF * (LDS_A + LDS_B) : CROWS * CST;
constexpr int LDS_BYTES = (LDS_FLOATS + 4) * 4;

constexpr int NTILES  = (GM / BM) * (NL / BN);
constexpr int NCHUNK  = 8;

constexpr int NP = 32;
constexpr int NG = 512;

constexpr size_t WS_CTRL   = 0;
constexpr size_t WS_TFLAG  = 64u << 10;
constexpr size_t WS_X      = 1u << 20;
constexpr size_t WS_W      = WS_X + (size_t)GM * GK * 4;
constexpr size_t WS_END    = WS_W + (size_t)GK * NL * 4;

constexpr int LINE = 64;
constexpr int C_TICKET = 0, C_TMO = 1, C_TICKET_G = 2;
constexpr int C_IN   = 8;
constexpr int C_DONE = 24;

constexpr unsigned long long DEADLINE_TICKS = 100000000ull;

typedef float f32x16 __attribute__((ext_vector_type(16)));
typedef float f32x4 __attribute__((ext_vector_type(4)));

struct Params {
    const float* x;
    const float* w;
    float* out[NR];
    char*  ws[NR];
    int rank;
};

__device__ __forceinline__ unsigned* ctrl(char* ws, int idx) {
    return reinterpret_cast<unsigned*>(ws + WS_CTRL) + idx * LINE;
}

template <int SCOPE>
__device__ __forceinline__ bool wait_ge(unsigned* flag, unsigned target, unsigned* tmo, unsigned call) {
    unsigned long long t0 = __builtin_amdgcn_s_memrealtime();
    while (__hip_atomic_load(flag, __ATOMIC_RELAXED, SCOPE) < target) {
        if (__hip_atomic_load(tmo, __ATOMIC_RELAXED, __HIP_MEMORY_SCOPE_AGENT) == call ||
            __builtin_amdgcn_s_memrealtime() - t0 > DEADLINE_TICKS) {
            __hip_atomic_store(tmo, call, __ATOMIC_RELAXED, __HIP_MEMORY_SCOPE_AGENT);
            return false;
        }
        __builtin_amdgcn_s_sleep(2);
    }
    return true;
}

__device__ __forceinline__ void tile_coords(int T, int& tm, int& tn, int& shell) {
    const int g = T >> 4, q = T & 15;
    int s = 0;
    while ((s + 1) * (s + 1) <= g) ++s;
    const int r = g - s * s;
    const int bi = r < s ? r : s;
    const int bj = r < s ? s : r - s;
    tm = bi * 4 + (q >> 2);
    tn = bj * 4 + (q & 3);
    shell = s;
}

__global__ void __launch_bounds__(256, 2) fused_kernel(Params p, int role) {
    extern __shared__ __attribute__((aligned(16))) float lds[];
    char* myws = p.ws[p.rank];
    const int tid = threadIdx.x;
    unsigned* s_words = reinterpret_cast<unsigned*>(lds) + (role == 0 ? 0 : LDS_FLOATS);

    if (tid == 0) {
        unsigned t = __hip_atomic_fetch_add(ctrl(myws, role == 0 ? C_TICKET : C_TICKET_G), 1u, __ATOMIC_RELAXED, __HIP_MEMORY_SCOPE_AGENT);
        s_words[0] = t / gridDim.x + 1;
    }
    __syncthreads();
    const unsigned call = s_words[0];
    unsigned* tmo = ctrl(myws, C_TMO);
    unsigned* tflag = reinterpret_cast<unsigned*>(myws + WS_TFLAG);

    if (role == 0) {
        const int pw = blockIdx.x;
        for (int c = 0; c < NCHUNK; ++c) {
            for (int u0 = pw; u0 < 4096; u0 += 4 * NP) {
                const float* src[4]; float* dstp[4]; f32x4 v[4];
                #pragma unroll
                for (int j = 0; j < 4; ++j) {
                    const int u = u0 + j * NP;
                    const int dst = (p.rank + (u & 7)) & (NR - 1);
                    const int idx = u >> 4;
                    char* dws = p.ws[dst];
                    const int r2 = 2 * idx + (tid >> 7), c4 = (tid & 127) * 4;
                    if (((u >> 3) & 1) == 0) {
                        const int row = c * 512 + r2;
                        src[j] = p.x + (size_t)row * KS + c4;
                        dstp[j] = reinterpret_cast<float*>(dws + WS_X) + (size_t)row * GK + p.rank * KS + c4;
                    } else {
                        src[j] = p.w + (size_t)r2 * GN + (size_t)dst * NL + c * 512 + c4;
                        dstp[j] = reinterpret_cast<float*>(dws + WS_W) + (size_t)(p.rank * KS + r2) * NL + c * 512 + c4;
                    }
                }
                #pragma unroll
                for (int j = 0; j < 4; ++j) v[j] = *reinterpret_cast<const f32x4*>(src[j]);
                #pragma unroll
                for (int j = 0; j < 4; ++j) *reinterpret_cast<f32x4*>(dstp[j]) = v[j];
            }
            asm volatile("s_waitcnt vmcnt(0)" ::: "memory");
            __syncthreads();
            if (tid == 0) {
                __builtin_amdgcn_fence(__ATOMIC_RELEASE, "");
                asm volatile("s_waitcnt vmcnt(0)" ::: "memory");
                #pragma unroll
                for (int q = 0; q < NR; ++q) {
                    const int d = (p.rank + q) & (NR - 1);
                    __hip_atomic_fetch_add(ctrl(p.ws[d], C_IN + c), 1u, __ATOMIC_RELAXED, __HIP_MEMORY_SCOPE_SYSTEM);
                }
            }
        }
        const int c4 = tid & 31, r0 = tid >> 5;
        for (int T = pw; T < NTILES; T += NP) {
            if (tid == 0) {
                wait_ge<__HIP_MEMORY_SCOPE_AGENT>(tflag + T, call, tmo, call);
                __builtin_amdgcn_fence(__ATOMIC_ACQUIRE, "agent");
                asm volatile("s_waitcnt vmcnt(0)" ::: "memory");
            }
            __syncthreads();
            int tm, tn, sh;
            tile_coords(T, tm, tn, sh);
            #pragma unroll 4
            for (int i = 0; i < BM / 8; ++i) {
                const int row = tm * BM + r0 + 8 * i;
                const size_t off = (size_t)row * GN + (size_t)p.rank * NL + tn * BN + c4 * 4;
                f32x4 v = *reinterpret_cast<const f32x4*>(p.out[p.rank] + off);
                #pragma unroll
                for (int q = 1; q < NR; ++q) {
                    const int d = (p.rank + q) & (NR - 1);
                    *reinterpret_cast<f32x4*>(p.out[d] + off) = v;
                }
            }
        }
        asm volatile("s_waitcnt vmcnt(0)" ::: "memory");
        __syncthreads();
        if (tid == 0) {
            __builtin_amdgcn_fence(__ATOMIC_RELEASE, "");
            asm volatile("s_waitcnt vmcnt(0)" ::: "memory");
            #pragma unroll
            for (int q = 1; q < NR; ++q) {
                const int d = (p.rank + q) & (NR - 1);
                __hip_atomic_fetch_add(ctrl(p.ws[d], C_DONE + p.rank), 1u, __ATOMIC_RELAXED, __HIP_MEMORY_SCOPE_SYSTEM);
            }
            if (pw == 0) {
                bool ok = true;
                for (int q = 1; q < NR; ++q) {
                    const int s = (p.rank + q) & (NR - 1);
                    ok = wait_ge<__HIP_MEMORY_SCOPE_SYSTEM>(ctrl(myws, C_DONE + s), (unsigned)NP * call, tmo, call) && ok;
                }
                __builtin_amdgcn_fence(__ATOMIC_ACQUIRE, "");
                if (!ok || __hip_atomic_load(tmo, __ATOMIC_RELAXED, __HIP_MEMORY_SCOPE_AGENT) == call)
                    p.out[p.rank][(size_t)p.rank * NL] = __builtin_nanf("");
            }
        }
        return;
    }

    const int gid = blockIdx.x;
    const int xl = gid & 7, slot = gid >> 3;
    const float* A = reinterpret_cast<const float*>(myws + WS_X);
    const float* B = reinterpret_cast<const float*>(myws + WS_W);
    const int wave = tid >> 6, lane = tid & 63;
    const int wm = wave >> 1, wn = wave & 1;
    const int l31 = lane & 31, h = lane >> 5;
    float* As = lds;
    float* Bs = lds + NBUF * LDS_A;
    char* ldsA = reinterpret_cast<char*>(As) + wave * 2048 + lane * 16;
    char* ldsB = reinterpret_cast<char*>(Bs) + wave * 2048 + lane * 16;
    const int arow = wm * 64 + l31;
    const float* fA = As + arow * BK;
    const float* fB = Bs + (4 * h) * BN + wn * 64 + l31;
    const int asw = (arow >> 2) & 3;
    constexpr int NT = GK / BK;

    for (int round = 0; round < NTILES / NG; ++round) {
        const int g = xl + 8 * (4 * round + (slot >> 4));
        const int T = g * 16 + (slot & 15);
        int tm, tn, shell;
        tile_coords(T, tm, tn, shell);
        const int m0 = tm * BM, n0 = tn * BN;

        if (tid == 0) {
            bool ok = wait_ge<__HIP_MEMORY_SCOPE_SYSTEM>(ctrl(myws, C_IN + shell), (unsigned)(NR * NP) * call, tmo, call);
            __builtin_amdgcn_fence(__ATOMIC_ACQUIRE, "");
            asm volatile("s_waitcnt vmcnt(0)" ::: "memory");
            ok = ok && (__hip_atomic_load(tmo, __ATOMIC_RELAXED, __HIP_MEMORY_SCOPE_AGENT) != call);
            s_words[1] = ok ? 1u : 0u;
        }
        __syncthreads();
        const bool ok = s_words[1] != 0;

        f32x16 acc[2][2];
        #pragma unroll
        for (int i = 0; i < 2; ++i)
            #pragma unroll
            for (int j = 0; j < 2; ++j)
                #pragma unroll
                for (int r = 0; r < 16; ++r) acc[i][j][r] = 0.f;

        const float* gA = A + (size_t)(m0 + wave * 32 + (lane >> 2)) * GK + (((lane & 3) ^ ((lane >> 4) & 3)) * 4);
        const float* gB = B + (size_t)(wave * 4 + (lane >> 5)) * NL + n0 + (lane & 31) * 4;

        if (ok) {
#define STAGE(KT, BUF) do { const float* pa = gA + (KT) * BK; char* la = ldsA + (BUF) * (LDS_A * 4); \
            const float* pb = gB + (size_t)(KT) * BK * NL; char* lb = ldsB + (BUF) * (LDS_B * 4); \
            __builtin_amdgcn_global_load_lds((const unsigned*)(pa), (unsigned*)(la), 16, 0, 0); \
            __builtin_amdgcn_global_load_lds((const unsigned*)(pa + 16 * GK), (unsigned*)(la + 1024), 16, 0, 0); \
            __builtin_amdgcn_global_load_lds((const unsigned*)(pb), (unsigned*)(lb), 16, 0, 0); \
            __builtin_amdgcn_global_load_lds((const unsigned*)(pb + 2 * NL), (unsigned*)(lb + 1024), 16, 0, 0); } while (0)
            STAGE(0, 0);
            STAGE(1, 1);
            STAGE(2, 2);
            asm volatile("s_waitcnt vmcnt(8)" ::: "memory");
            __builtin_amdgcn_s_barrier();
            f32x4 afA0, afA1, afB0, afB1;
            float bA[8], bB[8];
#define SB __builtin_amdgcn_sched_barrier(0)
#define RD_AF(AF0, AF1, BUF, KG) do { const float* a_ = fA + (BUF) * LDS_A + (((2 * (KG) + h) ^ asw) * 4); \
            AF0 = *reinterpret_cast<const f32x4*>(a_); AF1 = *reinterpret_cast<const f32x4*>(a_ + 32 * BK); } while (0)
#define RD_BLO(BF, BUF, KG) do { const float* b_ = fB + (BUF) * LDS_B + (KG) * 8 * BN; \
            BF[0] = b_[0]; BF[1] = b_[32]; BF[2] = b_[BN]; BF[3] = b_[BN + 32]; } while (0)
#define RD_BHI(BF, BUF, KG) do { const float* b_ = fB + (BUF) * LDS_B + (KG) * 8 * BN; \
            BF[4] = b_[2 * BN]; BF[5] = b_[2 * BN + 32]; BF[6] = b_[3 * BN]; BF[7] = b_[3 * BN + 32]; } while (0)
#define MF4(AF0, AF1, AV, B0, B1) \
            acc[0][0] = __builtin_amdgcn_mfma_f32_32x32x2f32(AF0.AV, B0, acc[0][0], 0, 0, 0); \
            acc[0][1] = __builtin_amdgcn_mfma_f32_32x32x2f32(AF0.AV, B1, acc[0][1], 0, 0, 0); \
            acc[1][0] = __builtin_amdgcn_mfma_f32_32x32x2f32(AF1.AV, B0, acc[1][0], 0, 0, 0); \
            acc[1][1] = __builtin_amdgcn_mfma_f32_32x32x2f32(AF1.AV, B1, acc[1][1], 0, 0, 0);
            RD_AF(afA0, afA1, 0, 0); RD_BLO(bA, 0, 0); RD_BHI(bA, 0, 0);
#define KSTEP(KT, BUF) do { if ((KT) < NT) { \
            MF4(afA0, afA1, x, bA[0], bA[1]) SB; RD_AF(afB0, afB1, BUF, 1); SB; \
            MF4(afA0, afA1, y, bA[2], bA[3]) SB; RD_BLO(bB, BUF, 1); SB; \
            MF4(afA0, afA1, z, bA[4], bA[5]) SB; RD_BHI(bB, BUF, 1); SB; \
            MF4(afA0, afA1, w, bA[6], bA[7]) SB; \
            if ((KT) + 2 < NT) { asm volatile("s_waitcnt vmcnt(4)" ::: "memory"); } \
            else { asm volatile("s_waitcnt vmcnt(0)" ::: "memory"); } \
            __builtin_amdgcn_s_barrier(); SB; \
            if ((KT) + 3 < NT) STAGE((KT) + 3, ((BUF) + 3) & 3); SB; \
            MF4(afB0, afB1, x, bB[0], bB[1]) SB; if ((KT) + 1 < NT) RD_AF(afA0, afA1, ((BUF) + 1) & 3, 0); SB; \
            MF4(afB0, afB1, y, bB[2], bB[3]) SB; if ((KT) + 1 < NT) RD_BLO(bA, ((BUF) + 1) & 3, 0); SB; \
            MF4(afB0, afB1, z, bB[4], bB[5]) SB; if ((KT) + 1 < NT) RD_BHI(bA, ((BUF) + 1) & 3, 0); SB; \
            MF4(afB0, afB1, w, bB[6], bB[7]) SB; \
            } } while (0)
            for (int kt = 0; kt < NT; kt += 4) {
                KSTEP(kt, 0);
                KSTEP(kt + 1, 1);
                KSTEP(kt + 2, 2);
                KSTEP(kt + 3, 3);
            }
        }

        float* Cs = lds;
        #pragma unroll
        for (int half = 0; half < 2; ++half) {
            __syncthreads();
            if (wm == half) {
                #pragma unroll
                for (int bi = 0; bi < 2; ++bi)
                    #pragma unroll
                    for (int bj = 0; bj < 2; ++bj)
                        #pragma unroll
                        for (int r = 0; r < 16; ++r) {
                            const int row = bi * 32 + (r & 3) + 8 * (r >> 2) + 4 * h;
                            const int col = wn * 64 + bj * 32 + l31;
                            const float v = acc[bi][bj][r];
                            Cs[row * CST + col] = ok ? fmaxf(v, 0.f) : __builtin_nanf("");
                        }
            }
            __syncthreads();
            const int c4 = tid & 31, r0 = tid >> 5;
            #pragma unroll
            for (int i = 0; i < CROWS / 8; ++i) {
                const int row = r0 + 8 * i;
                f32x4 v = *reinterpret_cast<const f32x4*>(Cs + row * CST + c4 * 4);
                const size_t off = (size_t)(m0 + half * 64 + row) * GN + (size_t)p.rank * NL + n0 + c4 * 4;
                *reinterpret_cast<f32x4*>(p.out[p.rank] + off) = v;
            }
        }
        asm volatile("s_waitcnt vmcnt(0)" ::: "memory");
        __syncthreads();
        if (tid == 0) {
            __builtin_amdgcn_fence(__ATOMIC_RELEASE, "agent");
            asm volatile("s_waitcnt vmcnt(0)" ::: "memory");
            __hip_atomic_store(tflag + T, call, __ATOMIC_RELAXED, __HIP_MEMORY_SCOPE_AGENT);
        }
    }
}

}

extern "C" void kernel_launch(int rank, int nranks, ncclComm_t comm,
                              hipStream_t stream,
                              void* const* d_in, const int* in_sizes, int n_in,
                              void* d_out, int out_size,
                              const int* in_dtypes, int out_dtype,
                              void* const* d_out_all,
                              void* const* d_ws_all, size_t ws_bytes) {
    (void)comm; (void)in_sizes; (void)n_in; (void)out_size; (void)in_dtypes; (void)out_dtype; (void)d_out;
    if (nranks != NR || ws_bytes < WS_END) {
        fprintf(stderr, "kernel_launch: unsupported configuration (nranks=%d ws=%zu)\n", nranks, ws_bytes);
        return;
    }
    Params p;
    p.x = static_cast<const float*>(d_in[0]);
    p.w = static_cast<const float*>(d_in[1]);
    for (int i = 0; i < NR; ++i) {
        p.out[i] = static_cast<float*>(d_out_all[i]);
        p.ws[i]  = static_cast<char*>(d_ws_all[i]);
    }
    p.rank = rank;
    if (hipFuncSetAttribute(reinterpret_cast<const void*>(fused_kernel),
                            hipFuncAttributeMaxDynamicSharedMemorySize, LDS_BYTES) != hipSuccess) {
        fprintf(stderr, "kernel_launch: hipFuncSetAttribute failed\n");
        return;
    }
    static hipStream_t side = nullptr;
    static hipEvent_t ev_fork = nullptr, ev_join = nullptr;
    if (!side) {
        if (hipStreamCreateWithFlags(&side, hipStreamNonBlocking) != hipSuccess ||
            hipEventCreateWithFlags(&ev_fork, hipEventDisableTiming) != hipSuccess ||
            hipEventCreateWithFlags(&ev_join, hipEventDisableTiming) != hipSuccess) {
            fprintf(stderr, "kernel_launch: stream/event creation failed\n");
            return;
        }
    }
    hipEventRecord(ev_fork, stream);
    hipStreamWaitEvent(side, ev_fork, 0);
    fused_kernel<<<NP, 256, 64, stream>>>(p, 0);
    fused_kernel<<<NG, 256, LDS_BYTES, side>>>(p, 1);
    hipEventRecord(ev_join, side);
    hipStreamWaitEvent(stream, ev_join, 0);
}
```

```cpp
#include <hip/hip_runtime.h>
#include <rccl/rccl.h>
#include <cstdio>
#include <cstdint>

namespace {

constexpr int NR   = 8;
constexpr int GM   = 4096;
constexpr int GK   = 4096;
constexpr int GN   = 32768;
constexpr int KS   = GK / NR;
constexpr int NL   = GN / NR;

constexpr int BM = 128, BN = 128, BK = 16;
constexpr int LDS_A = BM * BK;
constexpr int LDS_B = BK * BN;
constexpr int NBUF = 4;

constexpr int CST = BN + 4;
constexpr int CROWS = 64;
constexpr int LDS_FLOATS = (NBUF * (LDS_A + LDS_B) > CROWS * CST) ? NBUF * (LDS_A + LDS_B) : CROWS * CST;
constexpr int LDS_BYTES = (LDS_FLOATS + 4) * 4;

constexpr int NTILES  = (GM / BM) * (NL / BN);
constexpr int NCHUNK  = 8;

constexpr int NP = 32;
constexpr int NG = 512;

constexpr size_t WS_CTRL   = 0;
constexpr size_t WS_TFLAG  = 64u << 10;
constexpr size_t WS_X      = 1u << 20;
constexpr size_t WS_W      = WS_X + (size_t)GM * GK * 4;
constexpr size_t WS_END    = WS_W + (size_t)GK * NL * 4;

constexpr int LINE = 64;
constexpr int C_TICKET = 0, C_TMO = 1, C_TICKET_G = 2;
constexpr int C_IN   = 8;
constexpr int C_DONE = 24;

constexpr unsigned long long DEADLINE_TICKS = 100000000ull;

typedef float f32x16 __attribute__((ext_vector_type(16)));
typedef float f32x4 __attribute__((ext_vector_type(4)));

struct Params {
    const float* x;
    const float* w;
    float* out[NR];
    char*  ws[NR];
    int rank;
};

__device__ __forceinline__ unsigned* ctrl(char* ws, int idx) {
    return reinterpret_cast<unsigned*>(ws + WS_CTRL) + idx * LINE;
}

template <int SCOPE>
__device__ __forceinline__ bool wait_ge(unsigned* flag, unsigned target, unsigned* tmo, unsigned call) {
    unsigned long long t0 = __builtin_amdgcn_s_memrealtime();
    while (__hip_atomic_load(flag, __ATOMIC_RELAXED, SCOPE) < target) {
        if (__hip_atomic_load(tmo, __ATOMIC_RELAXED, __HIP_MEMORY_SCOPE_AGENT) == call ||
            __builtin_amdgcn_s_memrealtime() - t0 > DEADLINE_TICKS) {
            __hip_atomic_store(tmo, call, __ATOMIC_RELAXED, __HIP_MEMORY_SCOPE_AGENT);
            return false;
        }
        __builtin_amdgcn_s_sleep(2);
    }
    return true;
}

__device__ __forceinline__ void tile_coords(int T, int& tm, int& tn, int& shell) {
    const int g = T >> 4, q = T & 15;
    int s = 0;
    while ((s + 1) * (s + 1) <= g) ++s;
    const int r = g - s * s;
    const int bi = r < s ? r : s;
    const int bj = r < s ? s : r - s;
    tm = bi * 4 + (q >> 2);
    tn = bj * 4 + (q & 3);
    shell = s;
}

__global__ void __launch_bounds__(256, 2) fused_kernel(Params p, int role) {
    extern __shared__ __attribute__((aligned(16))) float lds[];
    char* myws = p.ws[p.rank];
    const int tid = threadIdx.x;
    unsigned* s_words = reinterpret_cast<unsigned*>(lds) + (role == 0 ? 0 : LDS_FLOATS);

    if (tid == 0) {
        unsigned t = __hip_atomic_fetch_add(ctrl(myws, role == 0 ? C_TICKET : C_TICKET_G), 1u, __ATOMIC_RELAXED, __HIP_MEMORY_SCOPE_AGENT);
        s_words[0] = t / gridDim.x + 1;
    }
    __syncthreads();
    const unsigned call = s_words[0];
    unsigned* tmo = ctrl(myws, C_TMO);
    unsigned* tflag = reinterpret_cast<unsigned*>(myws + WS_TFLAG);

    if (role == 0) {
        const int pw = blockIdx.x;
        for (int c = 0; c < NCHUNK; ++c) {
            for (int u0 = pw; u0 < 4096; u0 += 4 * NP) {
                const float* src[4]; float* dstp[4]; f32x4 v[4];
                #pragma unroll
                for (int j = 0; j < 4; ++j) {
                    const int u = u0 + j * NP;
                    const int dst = (p.rank + (u & 7)) & (NR - 1);
                    const int idx = u >> 4;
                    char* dws = p.ws[dst];
                    const int r2 = 2 * idx + (tid >> 7), c4 = (tid & 127) * 4;
                    if (((u >> 3) & 1) == 0) {
                        const int row = c * 512 + r2;
                        src[j] = p.x + (size_t)row * KS + c4;
                        dstp[j] = reinterpret_cast<float*>(dws + WS_X) + (size_t)row * GK + p.rank * KS + c4;
                    } else {
                        src[j] = p.w + (size_t)r2 * GN + (size_t)dst * NL + c * 512 + c4;
                        dstp[j] = reinterpret_cast<float*>(dws + WS_W) + (size_t)(p.rank * KS + r2) * NL + c * 512 + c4;
                    }
                }
                #pragma unroll
                for (int j = 0; j < 4; ++j) v[j] = *reinterpret_cast<const f32x4*>(src[j]);
                #pragma unroll
                for (int j = 0; j < 4; ++j) *reinterpret_cast<f32x4*>(dstp[j]) = v[j];
            }
            asm volatile("s_waitcnt vmcnt(0)" ::: "memory");
            __syncthreads();
            if (tid == 0) {
                __builtin_amdgcn_fence(__ATOMIC_RELEASE, "");
                asm volatile("s_waitcnt vmcnt(0)" ::: "memory");
                #pragma unroll
                for (int q = 0; q < NR; ++q) {
                    const int d = (p.rank + q) & (NR - 1);
                    __hip_atomic_fetch_add(ctrl(p.ws[d], C_IN + c), 1u, __ATOMIC_RELAXED, __HIP_MEMORY_SCOPE_SYSTEM);
                }
            }
        }
        const int c4 = tid & 31, r0 = tid >> 5;
        for (int T = pw; T < NTILES; T += NP) {
            if (tid == 0) {
                wait_ge<__HIP_MEMORY_SCOPE_AGENT>(tflag + T, call, tmo, call);
                __builtin_amdgcn_fence(__ATOMIC_ACQUIRE, "agent");
                asm volatile("s_waitcnt vmcnt(0)" ::: "memory");
            }
            __syncthreads();
            int tm, tn, sh;
            tile_coords(T, tm, tn, sh);
            #pragma unroll 4
            for (int i = 0; i < BM / 8; ++i) {
                const int row = tm * BM + r0 + 8 * i;
                const size_t off = (size_t)row * GN + (size_t)p.rank * NL + tn * BN + c4 * 4;
                f32x4 v = *reinterpret_cast<const f32x4*>(p.out[p.rank] + off);
                #pragma unroll
                for (int q = 1; q < NR; ++q) {
                    const int d = (p.rank + q) & (NR - 1);
                    *reinterpret_cast<f32x4*>(p.out[d] + off) = v;
                }
            }
        }
        asm volatile("s_waitcnt vmcnt(0)" ::: "memory");
        __syncthreads();
        if (tid == 0) {
            __builtin_amdgcn_fence(__ATOMIC_RELEASE, "");
            asm volatile("s_waitcnt vmcnt(0)" ::: "memory");
            #pragma unroll
            for (int q = 1; q < NR; ++q) {
                const int d = (p.rank + q) & (NR - 1);
                __hip_atomic_fetch_add(ctrl(p.ws[d], C_DONE + p.rank), 1u, __ATOMIC_RELAXED, __HIP_MEMORY_SCOPE_SYSTEM);
            }
            if (pw == 0) {
                bool ok = true;
                for (int q = 1; q < NR; ++q) {
                    const int s = (p.rank + q) & (NR - 1);
                    ok = wait_ge<__HIP_MEMORY_SCOPE_SYSTEM>(ctrl(myws, C_DONE + s), (unsigned)NP * call, tmo, call) && ok;
                }
                __builtin_amdgcn_fence(__ATOMIC_ACQUIRE, "");
                if (!ok || __hip_atomic_load(tmo, __ATOMIC_RELAXED, __HIP_MEMORY_SCOPE_AGENT) == call)
                    p.out[p.rank][(size_t)p.rank * NL] = __builtin_nanf("");
            }
        }
        return;
    }

    const int gid = blockIdx.x;
    const int xl = gid & 7, slot = gid >> 3;
    const float* A = reinterpret_cast<const float*>(myws + WS_X);
    const float* B = reinterpret_cast<const float*>(myws + WS_W);
    const int wave = tid >> 6, lane = tid & 63;
    const int wm = wave >> 1, wn = wave & 1;
    const int l31 = lane & 31, h = lane >> 5;
    float* As = lds;
    float* Bs = lds + NBUF * LDS_A;
    char* ldsA = reinterpret_cast<char*>(As) + wave * 2048 + lane * 16;
    char* ldsB = reinterpret_cast<char*>(Bs) + wave * 2048 + lane * 16;
    const int arow = wm * 64 + l31;
    const float* fA = As + arow * BK;
    const float* fB = Bs + (4 * h) * BN + wn * 64 + l31;
    const int asw = (arow >> 2) & 3;
    constexpr int NT = GK / BK;

    for (int round = 0; round < NTILES / NG; ++round) {
        const int g = xl + 8 * (4 * round + (slot >> 4));
        const int T = g * 16 + (slot & 15);
        int tm, tn, shell;
        tile_coords(T, tm, tn, shell);
        const int m0 = tm * BM, n0 = tn * BN;

        if (tid == 0) {
            bool ok = wait_ge<__HIP_MEMORY_SCOPE_SYSTEM>(ctrl(myws, C_IN + shell), (unsigned)(NR * NP) * call, tmo, call);
            __builtin_amdgcn_fence(__ATOMIC_ACQUIRE, "");
            asm volatile("s_waitcnt vmcnt(0)" ::: "memory");
            ok = ok && (__hip_atomic_load(tmo, __ATOMIC_RELAXED, __HIP_MEMORY_SCOPE_AGENT) != call);
            s_words[1] = ok ? 1u : 0u;
        }
        __syncthreads();
        const bool ok = s_words[1] != 0;

        f32x16 acc[2][2];
        #pragma unroll
        for (int i = 0; i < 2; ++i)
            #pragma unroll
            for (int j = 0; j < 2; ++j)
                #pragma unroll
                for (int r = 0; r < 16; ++r) acc[i][j][r] = 0.f;

        const float* gA = A + (size_t)(m0 + wave * 32 + (lane >> 2)) * GK + (((lane & 3) ^ ((lane >> 4) & 3)) * 4);
        const float* gB = B + (size_t)(wave * 4 + (lane >> 5)) * NL + n0 + (lane & 31) * 4;

        if (ok) {
#define STAGE(KT, BUF) do { const float* pa = gA + (KT) * BK; char* la = ldsA + (BUF) * (LDS_A * 4); \
            const float* pb = gB + (size_t)(KT) * BK * NL; char* lb = ldsB + (BUF) * (LDS_B * 4); \
            __builtin_amdgcn_global_load_lds((const unsigned*)(pa), (unsigned*)(la), 16, 0, 0); \
            __builtin_amdgcn_global_load_lds((const unsigned*)(pa + 16 * GK), (unsigned*)(la + 1024), 16, 0, 0); \
            __builtin_amdgcn_global_load_lds((const unsigned*)(pb), (unsigned*)(lb), 16, 0, 0); \
            __builtin_amdgcn_global_load_lds((const unsigned*)(pb + 2 * NL), (unsigned*)(lb + 1024), 16, 0, 0); } while (0)
            STAGE(0, 0);
            STAGE(1, 1);
            STAGE(2, 2);
            asm volatile("s_waitcnt vmcnt(8)" ::: "memory");
            __builtin_amdgcn_s_barrier();
            f32x4 afA0, afA1, afB0, afB1;
            float bA[8], bB[8];
#define SB __builtin_amdgcn_sched_barrier(0)
#define RD_AF(AF0, AF1, BUF, KG) do { const float* a_ = fA + (BUF) * LDS_A + (((2 * (KG) + h) ^ asw) * 4); \
            AF0 = *reinterpret_cast<const f32x4*>(a_); AF1 = *reinterpret_cast<const f32x4*>(a_ + 32 * BK); } while (0)
#define RD_BLO(BF, BUF, KG) do { const float* b_ = fB + (BUF) * LDS_B + (KG) * 8 * BN; \
            BF[0] = b_[0]; BF[1] = b_[32]; BF[2] = b_[BN]; BF[3] = b_[BN + 32]; } while (0)
#define RD_BHI(BF, BUF, KG) do { const float* b_ = fB + (BUF) * LDS_B + (KG) * 8 * BN; \
            BF[4] = b_[2 * BN]; BF[5] = b_[2 * BN + 32]; BF[6] = b_[3 * BN]; BF[7] = b_[3 * BN + 32]; } while (0)
#define MF4(AF0, AF1, AV, B0, B1) \
            acc[0][0] = __builtin_amdgcn_mfma_f32_32x32x2f32(AF0.AV, B0, acc[0][0], 0, 0, 0); \
            acc[0][1] = __builtin_amdgcn_mfma_f32_32x32x2f32(AF0.AV, B1, acc[0][1], 0, 0, 0); \
            acc[1][0] = __builtin_amdgcn_mfma_f32_32x32x2f32(AF1.AV, B0, acc[1][0], 0, 0, 0); \
            acc[1][1] = __builtin_amdgcn_mfma_f32_32x32x2f32(AF1.AV, B1, acc[1][1], 0, 0, 0);
            RD_AF(afA0, afA1, 0, 0); RD_BLO(bA, 0, 0); RD_BHI(bA, 0, 0);
#define KSTEP(KT, BUF) do { if ((KT) < NT) { \
            MF4(afA0, afA1, x, bA[0], bA[1]) SB; RD_AF(afB0, afB1, BUF, 1); SB; \
            MF4(afA0, afA1, y, bA[2], bA[3]) SB; RD_BLO(bB, BUF, 1); SB; \
            MF4(afA0, afA1, z, bA[4], bA[5]) SB; RD_BHI(bB, BUF, 1); SB; \
            MF4(afA0, afA1, w, bA[6], bA[7]) SB; \
            if ((KT) + 2 < NT) { asm volatile("s_waitcnt vmcnt(4)" ::: "memory"); } \
            else { asm volatile("s_waitcnt vmcnt(0)" ::: "memory"); } \
            __builtin_amdgcn_s_barrier(); SB; \
            if ((KT) + 3 < NT) STAGE((KT) + 3, ((BUF) + 3) & 3); SB; \
            MF4(afB0, afB1, x, bB[0], bB[1]) SB; if ((KT) + 1 < NT) RD_AF(afA0, afA1, ((BUF) + 1) & 3, 0); SB; \
            MF4(afB0, afB1, y, bB[2], bB[3]) SB; if ((KT) + 1 < NT) RD_BLO(bA, ((BUF) + 1) & 3, 0); SB; \
            MF4(afB0, afB1, z, bB[4], bB[5]) SB; if ((KT) + 1 < NT) RD_BHI(bA, ((BUF) + 1) & 3, 0); SB; \
            MF4(afB0, afB1, w, bB[6], bB[7]) SB; \
            } } while (0)
            for (int kt = 0; kt < NT; kt += 4) {
                KSTEP(kt, 0);
                KSTEP(kt + 1, 1);
                KSTEP(kt + 2, 2);
                KSTEP(kt + 3, 3);
            }
        }

        float* Cs = lds;
        #pragma unroll
        for (int half = 0; half < 2; ++half) {
            __syncthreads();
            if (wm == half) {
                #pragma unroll
                for (int bi = 0; bi < 2; ++bi)
                    #pragma unroll
                    for (int bj = 0; bj < 2; ++bj)
                        #pragma unroll
                        for (int r = 0; r < 16; ++r) {
                            const int row = bi * 32 + (r & 3) + 8 * (r >> 2) + 4 * h;
                            const int col = wn * 64 + bj * 32 + l31;
                            const float v = acc[bi][bj][r];
                            Cs[row * CST + col] = ok ? fmaxf(v, 0.f) : __builtin_nanf("");
                        }
            }
            __syncthreads();
            const int c4 = tid & 31, r0 = tid >> 5;
            #pragma unroll
            for (int i = 0; i < CROWS / 8; ++i) {
                const int row = r0 + 8 * i;
                f32x4 v = *reinterpret_cast<const f32x4*>(Cs + row * CST + c4 * 4);
                const size_t off = (size_t)(m0 + half * 64 + row) * GN + (size_t)p.rank * NL + n0 + c4 * 4;
                *reinterpret_cast<f32x4*>(p.out[p.rank] + off) = v;
            }
        }
        asm volatile("s_waitcnt vmcnt(0)" ::: "memory");
        __syncthreads();
        if (tid == 0) {
            __builtin_amdgcn_fence(__ATOMIC_RELEASE, "agent");
            asm volatile("s_waitcnt vmcnt(0)" ::: "memory");
            __hip_atomic_store(tflag + T, call, __ATOMIC_RELAXED, __HIP_MEMORY_SCOPE_AGENT);
        }
    }
}

}

extern "C" void kernel_launch(int rank, int nranks, ncclComm_t comm,
                              hipStream_t stream,
                              void* const* d_in, const int* in_sizes, int n_in,
                              void* d_out, int out_size,
                              const int* in_dtypes, int out_dtype,
                              void* const* d_out_all,
                              void* const* d_ws_all, size_t ws_bytes) {
    (void)comm; (void)in_sizes; (void)n_in; (void)out_size; (void)in_dtypes; (void)out_dtype; (void)d_out;
    if (nranks != NR || ws_bytes < WS_END) {
        fprintf(stderr, "kernel_launch: unsupported configuration (nranks=%d ws=%zu)\n", nranks, ws_bytes);
        return;
    }
    Params p;
    p.x = static_cast<const float*>(d_in[0]);
    p.w = static_cast<const float*>(d_in[1]);
    for (int i = 0; i < NR; ++i) {
        p.out[i] = static_cast<float*>(d_out_all[i]);
        p.ws[i]  = static_cast<char*>(d_ws_all[i]);
    }
    p.rank = rank;
    if (hipFuncSetAttribute(reinterpret_cast<const void*>(fused_kernel),
                            hipFuncAttributeMaxDynamicSharedMemorySize, LDS_BYTES) != hipSuccess) {
        fprintf(stderr, "kernel_launch: hipFuncSetAttribute failed\n");
        return;
    }
    static hipStream_t side_tab[NR] = {};
    static hipEvent_t fork_tab[NR] = {}, join_tab[NR] = {};
    hipStream_t& side = side_tab[rank];
    hipEvent_t& ev_fork = fork_tab[rank];
    hipEvent_t& ev_join = join_tab[rank];
    if (!side) {
        if (hipStreamCreateWithFlags(&side, hipStreamNonBlocking) != hipSuccess ||
            hipEventCreateWithFlags(&ev_fork, hipEventDisableTiming) != hipSuccess ||
            hipEventCreateWithFlags(&ev_join, hipEventDisableTiming) != hipSuccess) {
            fprintf(stderr, "kernel_launch: stream/event creation failed\n");
            return;
        }
    }
    hipEventRecord(ev_fork, stream);
    hipStreamWaitEvent(side, ev_fork, 0);
    fused_kernel<<<NP, 256, 64, stream>>>(p, 0);
    fused_kernel<<<NG, 256, LDS_BYTES, side>>>(p, 1);
    hipEventRecord(ev_join, side);
    hipStreamWaitEvent(stream, ev_join, 0);
}
```

```cpp
#include <hip/hip_runtime.h>
#include <rccl/rccl.h>
#include <cstdio>
#include <cstdint>

namespace {

constexpr int NR = 8, GM = 4096, GK = 4096, GN = 32768, KS = GK / NR;

constexpr int BM = 256, BK = 64, HALF = 128, HTB = HALF * BK * 2, STAGE_BYTES = 8 * HTB;
constexpr int LDS_BYTES = STAGE_BYTES + 64;
constexpr int NP  = 16;
constexpr int NGW = 256;
constexpr int NWCH = 16;
constexpr int QLEN = 256;

constexpr size_t XH_BYTES = (size_t)GM * GK * 2;
constexpr size_t WH_BYTES = (size_t)GN * GK * 2;
constexpr size_t BUF_BYTES = XH_BYTES + WH_BYTES;
constexpr size_t WS_BUF = 1u << 20;
constexpr size_t WS_END = WS_BUF + 2 * BUF_BYTES;

constexpr int LINE = 64;
constexpr int C_TICKET = 0, C_TMO = 1, C_EXIT = 2;
constexpr int C_Q = 8;
constexpr int C_X = 16;
constexpr int C_W = 24;

constexpr unsigned long long DEADLINE_TICKS = 100000000ull;

typedef float f32x4 __attribute__((ext_vector_type(4)));
typedef _Float16 f16x8 __attribute__((ext_vector_type(8)));
#define LAS __attribute__((address_space(3)))

struct Params {
    const float* x;
    const float* w;
    float* out;
    char*  ws[NR];
    int rank;
};

__device__ __forceinline__ unsigned* ctrl(char* ws, int idx) { return reinterpret_cast<unsigned*>(ws) + idx * LINE; }

__device__ __forceinline__ void st_wt(char* ptr, f16x8 v) {
    asm volatile("global_store_dwordx4 %0, %1, off sc0 sc1" :: "v"(ptr), "v"(v) : "memory");
}
__device__ __forceinline__ int lds_byte(int r, int c) { const int st = (r >> 4) * 2 + (c >> 5), rr = r & 15, cc = c & 31, ob = rr * 64 + cc * 2; return st * 1024 + (ob ^ (((ob >> 9) & 1) << 5)); }
__device__ __forceinline__ void stage_rc(int b, int& R, int& C) { const int st = b / 1024, sb = b % 1024, swz = sb ^ (((sb >> 9) & 1) << 5); R = (st >> 1) * 16 + swz / 64; C = (st & 1) * 32 + (swz % 64) / 2; }

__device__ __forceinline__ void tile_of(int t, int j, int& pm, int& pn) {
    const int i = t >> 5, q = t & 31;
    pm = 8 * (j & 1) + (q & 7);
    pn = 16 * i + 4 * (j >> 1) + (q >> 3);
}

__global__ void __launch_bounds__(512, 2) fused_kernel(Params p) {
    extern __shared__ __attribute__((aligned(16))) unsigned char shm[];
    LAS unsigned char* lds = (LAS unsigned char*)shm;
    char* myws = p.ws[p.rank];
    const int tid = threadIdx.x;
    LAS unsigned* s_words = (LAS unsigned*)(lds + STAGE_BYTES);

    if (tid == 0) {
        unsigned t = __hip_atomic_fetch_add(ctrl(myws, C_TICKET), 1u, __ATOMIC_RELAXED, __HIP_MEMORY_SCOPE_AGENT);
        s_words[0] = t / gridDim.x + 1;
        s_words[1] = 1u;
    }
    __syncthreads();
    const unsigned call = s_words[0];
    const size_t bufoff = WS_BUF + (size_t)(call & 1) * BUF_BYTES;

    if (blockIdx.x < NP) {
        const int pw = blockIdx.x;
        for (int c = -2; c < NWCH; ++c) {
            if (c < 0) {
                const int h = c + 2;
                #pragma unroll 4
                for (int it = 0; it < 2048 / NP / 8; ++it) {
                    const int row = h * 2048 + pw * (2048 / NP) + it * 8 + (tid >> 6), c8 = (tid & 63) * 8;
                    const f32x4 a = *reinterpret_cast<const f32x4*>(p.x + (size_t)row * KS + c8);
                    const f32x4 b = *reinterpret_cast<const f32x4*>(p.x + (size_t)row * KS + c8 + 4);
                    f16x8 v;
                    v[0] = (_Float16)a[0]; v[1] = (_Float16)a[1]; v[2] = (_Float16)a[2]; v[3] = (_Float16)a[3];
                    v[4] = (_Float16)b[0]; v[5] = (_Float16)b[1]; v[6] = (_Float16)b[2]; v[7] = (_Float16)b[3];
                    const size_t off = bufoff + ((size_t)row * GK + p.rank * KS + c8) * 2;
                    #pragma unroll
                    for (int q = 0; q < NR; ++q) st_wt(p.ws[(p.rank + q) & (NR - 1)] + off, v);
                }
            } else {
                for (int b = 0; b < 64 / NP; ++b) {
                    const int n0 = c * 2048 + (pw * (64 / NP) + b) * 32;
                    const int nq = tid & 7, kq = tid >> 3;
                    f32x4 r[8];
                    #pragma unroll
                    for (int j = 0; j < 8; ++j) r[j] = *reinterpret_cast<const f32x4*>(p.w + (size_t)(8 * kq + j) * GN + n0 + 4 * nq);
                    __syncthreads();
                    #pragma unroll
                    for (int i = 0; i < 4; ++i) {
                        f16x8 v;
                        #pragma unroll
                        for (int j = 0; j < 8; ++j) v[j] = (_Float16)r[j][i];
                        *(LAS f16x8*)(lds + (4 * nq + i) * 1040 + kq * 16) = v;
                    }
                    __syncthreads();
                    #pragma unroll
                    for (int j = 0; j < 4; ++j) {
                        const int idx = tid + 512 * j, n = idx >> 6, seg = idx & 63;
                        const f16x8 v = *(const LAS f16x8*)(lds + n * 1040 + seg * 16);
                        const size_t off = bufoff + XH_BYTES + ((size_t)(n0 + n) * GK + p.rank * KS) * 2 + seg * 16;
                        #pragma unroll
                        for (int q = 0; q < NR; ++q) st_wt(p.ws[(p.rank + q) & (NR - 1)] + off, v);
                    }
                }
            }
            asm volatile("s_waitcnt vmcnt(0)" ::: "memory");
            __syncthreads();
            if (tid == 0) {
                const int ci = c < 0 ? C_X + c + 2 : C_W + c;
                #pragma unroll
                for (int q = 0; q < NR; ++q)
                    __hip_atomic_fetch_add(ctrl(p.ws[(p.rank + q) & (NR - 1)], ci), 1u, __ATOMIC_RELAXED, __HIP_MEMORY_SCOPE_SYSTEM);
            }
        }
        return;
    }

    unsigned* tmo = ctrl(myws, C_TMO);
    const int xj = (int)((unsigned)__builtin_amdgcn_s_getreg((3 << 11) | 20) & 7u);
    unsigned* qhead = ctrl(myws, C_Q + xj);
    const unsigned need = (unsigned)(NR * NP) * call;
    const int wid = __builtin_amdgcn_readfirstlane(tid >> 6), lane = tid & 63, wr = wid >> 2, wc = wid & 3, fr = lane & 15, fq = lane >> 4;
    constexpr int nt = GK / BK;
    unsigned voff[2];
    #pragma unroll
    for (int i = 0; i < 2; ++i) { int R, C; stage_rc(tid * 16 + i * 8192, R, C); voff[i] = (unsigned)(R * GK + C) * 2u; }
    constexpr size_t kstep = BK * 2, hstep = (size_t)HALF * GK * 2, tstep = 2 * hstep;
    const unsigned ldsw = (unsigned)wid * 1024u;
    const int aoff = lds_byte(wr * 64 + fr, fq * 8), boff = lds_byte(wc * 32 + fr, fq * 8);
    const char* gA = myws + bufoff;
    const char* gB = myws + bufoff + XH_BYTES;
#define SA(b, h) (((b) * 2 + (h)) * HTB)
#define SB(b, h) ((4 + (b) * 2 + (h)) * HTB)
#define STAGE(bo, gbase) do { _Pragma("unroll") for (int _i = 0; _i < 2; ++_i) \
        __builtin_amdgcn_global_load_lds((const unsigned*)((const char*)(gbase) + voff[_i]), (LAS unsigned*)(lds + (bo) + ldsw + _i * 8192), 16, 0, 0); } while (0)
#define LDA(dst, b, h) do { _Pragma("unroll") for (int m = 0; m < 4; ++m) _Pragma("unroll") for (int k = 0; k < 2; ++k) dst[m][k] = *(const LAS f16x8*)(lds + SA(b, h) + aoff + m * 2048 + k * 1024); } while (0)
#define LDB(dst, b, h) do { _Pragma("unroll") for (int n = 0; n < 2; ++n) _Pragma("unroll") for (int k = 0; k < 2; ++k) dst[n][k] = *(const LAS f16x8*)(lds + SB(b, h) + boff + n * 2048 + k * 1024); } while (0)
#define MMA(ai, bj, At, Bt) do { __builtin_amdgcn_s_setprio(1); _Pragma("unroll") for (int m = 0; m < 4; ++m) _Pragma("unroll") for (int n = 0; n < 2; ++n) _Pragma("unroll") for (int k = 0; k < 2; ++k) \
        acc[ai][bj][m][n] = __builtin_amdgcn_mfma_f32_16x16x32_f16(Bt[n][k], At[m][k], acc[ai][bj][m][n], 0, 0, 0); __builtin_amdgcn_s_setprio(0); } while (0)
#define WAIT_V(n) asm volatile("s_waitcnt vmcnt(" #n ")" ::: "memory")
#define WAIT_L(n) asm volatile("s_waitcnt lgkmcnt(" #n ")" ::: "memory")
#define BAR __builtin_amdgcn_s_barrier()
#define SCHED __builtin_amdgcn_sched_barrier(0)
#define READY(PM, PN, ACQ) do { if (tid < 64) { \
        unsigned* f0 = ctrl(myws, C_X + ((PM) >> 3)); unsigned* f1 = ctrl(myws, C_W + ((PN) >> 3)); \
        const unsigned long long t0 = __builtin_amdgcn_s_memrealtime(); \
        while ((unsigned)__builtin_amdgcn_readfirstlane(__hip_atomic_load(f0, __ATOMIC_RELAXED, __HIP_MEMORY_SCOPE_SYSTEM)) < need || \
               (unsigned)__builtin_amdgcn_readfirstlane(__hip_atomic_load(f1, __ATOMIC_RELAXED, __HIP_MEMORY_SCOPE_SYSTEM)) < need) { \
            if ((unsigned)__builtin_amdgcn_readfirstlane(__hip_atomic_load(tmo, __ATOMIC_RELAXED, __HIP_MEMORY_SCOPE_AGENT)) == call || \
                __builtin_amdgcn_s_memrealtime() - t0 > DEADLINE_TICKS) { \
                if (tid == 0) { __hip_atomic_store(tmo, call, __ATOMIC_RELAXED, __HIP_MEMORY_SCOPE_AGENT); s_words[1] = 0u; } \
                break; } \
            __builtin_amdgcn_s_sleep(2); } \
        if (ACQ) __builtin_amdgcn_fence(__ATOMIC_ACQUIRE, ""); \
        asm volatile("s_waitcnt vmcnt(0) lgkmcnt(0)" ::: "memory"); } \
        asm volatile("" ::: "memory"); __builtin_amdgcn_s_barrier(); asm volatile("" ::: "memory"); } while (0)

    if (tid == 0) s_words[2] = __hip_atomic_fetch_add(qhead, 1u, __ATOMIC_RELAXED, __HIP_MEMORY_SCOPE_AGENT);
    __syncthreads();
    int cur = (int)s_words[2];
    if (cur < QLEN) {
        int pm, pn, ui = 0;
        tile_of(cur, xj, pm, pn);
        f32x4 acc[2][2][4][2];
        #pragma unroll
        for (int a = 0; a < 2; ++a)
            #pragma unroll
            for (int b = 0; b < 2; ++b)
                #pragma unroll
                for (int m = 0; m < 4; ++m)
                    #pragma unroll
                    for (int n = 0; n < 2; ++n) acc[a][b][m][n] = (f32x4){0.f, 0.f, 0.f, 0.f};
        f16x8 At[4][2], B0[2][2], B1[2][2];
        const char* cA = gA + (size_t)pm * tstep; const char* cB = gB + (size_t)pn * tstep;
        READY(pm, pn, true);
        STAGE(SB(0, 0), cB); STAGE(SA(0, 0), cA); STAGE(SB(0, 1), cB + hstep); STAGE(SA(0, 1), cA + hstep);
        if (wr == 1) BAR;
        WAIT_V(4); BAR;
        STAGE(SB(1, 0), cB + kstep); STAGE(SA(1, 0), cA + kstep); STAGE(SB(1, 1), cB + hstep + kstep);
        WAIT_V(6); BAR;
        for (;;) {
            bool has_next = false;
            int npm = pm, npn = pn;
            const char* nA = cA; const char* nB = cB;
            for (int t = 0; t < nt; t += 2) {
                const bool last = (t == nt - 2);
                if (t == 8 && tid == 0) s_words[2 + ((ui + 1) & 1)] = __hip_atomic_fetch_add(qhead, 1u, __ATOMIC_RELAXED, __HIP_MEMORY_SCOPE_AGENT);
                if (last) {
                    const int nx = (int)s_words[2 + ((ui + 1) & 1)];
                    has_next = nx < QLEN;
                    if (has_next) {
                        tile_of(nx, xj, npm, npn);
                        nA = gA + (size_t)npm * tstep; nB = gB + (size_t)npn * tstep;
                        READY(npm, npn, false);
                    }
                }
                const char* a1 = cA + (size_t)(t + 1) * kstep;
                const char* a2 = last ? nA : cA + (size_t)(t + 2) * kstep; const char* b2 = last ? nB : cB + (size_t)(t + 2) * kstep;
                const char* a3 = a2 + kstep; const char* b3 = b2 + kstep;
                LDB(B0, 0, 0); SCHED; LDA(At, 0, 0); STAGE(SA(1, 1), a1 + hstep);
                WAIT_L(8); BAR; WAIT_L(0); MMA(0, 0, At, B0); BAR; SCHED;
                LDB(B1, 0, 1); STAGE(SB(0, 0), b2);
                BAR; WAIT_L(0); MMA(0, 1, At, B1); BAR;
                LDA(At, 0, 1); STAGE(SA(0, 0), a2);
                BAR; WAIT_L(0); MMA(1, 0, At, B0); BAR; SCHED;
                STAGE(SB(0, 1), b2 + hstep);
                WAIT_V(6); BAR; MMA(1, 1, At, B1); BAR;
                LDB(B0, 1, 0); SCHED; LDA(At, 1, 0); STAGE(SA(0, 1), a2 + hstep);
                WAIT_L(8); BAR; WAIT_L(0); MMA(0, 0, At, B0); BAR; SCHED;
                LDB(B1, 1, 1); STAGE(SB(1, 0), b3);
                BAR; WAIT_L(0); MMA(0, 1, At, B1); BAR;
                LDA(At, 1, 1); STAGE(SA(1, 0), a3);
                BAR; WAIT_L(0); MMA(1, 0, At, B0); BAR; SCHED;
                STAGE(SB(1, 1), b3 + hstep);
                WAIT_V(6); BAR; MMA(1, 1, At, B1); BAR;
            }
            {
                const bool ok = s_words[1] != 0u;
                const float nanv = __builtin_nanf("");
                float* C0 = p.out + (size_t)(pm * BM + wr * 64 + fr) * GN + pn * BM + wc * 32 + 4 * fq;
                #pragma unroll
                for (int ai = 0; ai < 2; ++ai)
                    #pragma unroll
                    for (int m = 0; m < 4; ++m) {
                        float* rowp = C0 + (size_t)(ai * HALF + m * 16) * GN;
                        #pragma unroll
                        for (int bj = 0; bj < 2; ++bj)
                            #pragma unroll
                            for (int n = 0; n < 2; ++n) {
                                f32x4 v = acc[ai][bj][m][n];
                                #pragma unroll
                                for (int e = 0; e < 4; ++e) v[e] = ok ? fmaxf(v[e], 0.f) : nanv;

                                *reinterpret_cast<f32x4*>(rowp + bj * HALF + n * 16) = v;
                                acc[ai][bj][m][n] = (f32x4){0.f, 0.f, 0.f, 0.f};
                            }
                    }
            }
            if (!has_next) break;
            pm = npm; pn = npn; cA = nA; cB = nB; ++ui;
        }
        WAIT_V(0);
        if (wr == 0) BAR;
        BAR;
    }
    if (tid == 0) {
        const unsigned e = __hip_atomic_fetch_add(ctrl(myws, C_EXIT), 1u, __ATOMIC_ACQ_REL, __HIP_MEMORY_SCOPE_AGENT);
        if (e % NGW == NGW - 1) {
            #pragma unroll
            for (int j = 0; j < 8; ++j) __hip_atomic_store(ctrl(myws, C_Q + j), 0u, __ATOMIC_RELAXED, __HIP_MEMORY_SCOPE_AGENT);
        }
    }
}

}

extern "C" void kernel_launch(int rank, int nranks, ncclComm_t comm,
                              hipStream_t stream,
                              void* const* d_in, const int* in_sizes, int n_in,
                              void* d_out, int out_size,
                              const int* in_dtypes, int out_dtype,
                              void* const* d_out_all,
                              void* const* d_ws_all, size_t ws_bytes) {
    (void)comm; (void)in_sizes; (void)n_in; (void)out_size; (void)in_dtypes; (void)out_dtype; (void)d_out_all;
    if (nranks != NR || ws_bytes < WS_END) {
        fprintf(stderr, "kernel_launch: unsupported configuration (nranks=%d ws=%zu)\n", nranks, ws_bytes);
        return;
    }
    Params p;
    p.x = static_cast<const float*>(d_in[0]);
    p.w = static_cast<const float*>(d_in[1]);
    p.out = static_cast<float*>(d_out);
    for (int i = 0; i < NR; ++i) p.ws[i] = static_cast<char*>(d_ws_all[i]);
    p.rank = rank;
    if (hipFuncSetAttribute(reinterpret_cast<const void*>(fused_kernel),
                            hipFuncAttributeMaxDynamicSharedMemorySize, LDS_BYTES) != hipSuccess) {
        fprintf(stderr, "kernel_launch: hipFuncSetAttribute failed\n");
        return;
    }
    fused_kernel<<<NP + NGW, 512, LDS_BYTES, stream>>>(p);
}
```

```cpp
#include <hip/hip_runtime.h>
#include <rccl/rccl.h>
#include <cstdio>
#include <cstdint>

namespace {

constexpr int NR = 8, GM = 4096, GK = 4096, GN = 32768, KS = GK / NR;

constexpr int BM = 256, BK = 64, HALF = 128, HTB = HALF * BK * 2, STAGE_BYTES = 8 * HTB;
constexpr int LDS_BYTES = STAGE_BYTES + 64;
constexpr int NP  = 16;
constexpr int NGW = 256;
constexpr int U    = 2;
constexpr int NOWN = 8 * U;
constexpr int NWCH = 1 + (128 - NOWN) / 8;
constexpr int S0 = 5;
constexpr int QOWN = 2 * U;
constexpr int QLEN = QOWN + 2 * (128 - NOWN);

constexpr size_t XH_BYTES = (size_t)GM * GK * 2;
constexpr size_t WH_BYTES = (size_t)GN * GK * 2;
constexpr size_t BUF_BYTES = XH_BYTES + WH_BYTES;
constexpr size_t WS_BUF = 1u << 20;
constexpr size_t WS_END = WS_BUF + 2 * BUF_BYTES;

constexpr int LINE = 64;
constexpr int C_TICKET = 0, C_TMO = 1, C_EXIT = 2;
constexpr int C_Q = 8;
constexpr int C_X = 16;
constexpr int C_W = 24;
constexpr int C_DONE = 48;

constexpr unsigned long long DEADLINE_TICKS = 100000000ull;

typedef float f32x4 __attribute__((ext_vector_type(4)));
typedef _Float16 f16x8 __attribute__((ext_vector_type(8)));
#define LAS __attribute__((address_space(3)))

struct Params {
    const float* x;
    const float* w;
    float* out[NR];
    char*  ws[NR];
    int rank;
};

__device__ __forceinline__ unsigned* ctrl(char* ws, int idx) { return reinterpret_cast<unsigned*>(ws) + idx * LINE; }

__device__ __forceinline__ void st_wt(char* ptr, f16x8 v) {
    asm volatile("global_store_dwordx4 %0, %1, off sc0 sc1" :: "v"(ptr), "v"(v) : "memory");
}
__device__ __forceinline__ void st_wt32(float* ptr, f32x4 v) {
    asm volatile("global_store_dwordx4 %0, %1, off sc0 sc1" :: "v"(ptr), "v"(v) : "memory");
}
__device__ __forceinline__ int lds_byte(int r, int c) { const int st = (r >> 4) * 2 + (c >> 5), rr = r & 15, cc = c & 31, ob = rr * 64 + cc * 2; return st * 1024 + (ob ^ (((ob >> 9) & 1) << 5)); }
__device__ __forceinline__ void stage_rc(int b, int& R, int& C) { const int st = b / 1024, sb = b % 1024, swz = sb ^ (((sb >> 9) & 1) << 5); R = (st >> 1) * 16 + swz / 64; C = (st & 1) * 32 + (swz % 64) / 2; }

__device__ __forceinline__ void tile_of(int t, int j, int rank, int& pm, int& pn, int& ch) {
    if (t >= 32 * S0) {
        if (t < 32 * S0 + QOWN) { t -= 32 * S0; pm = 2 * j + (t & 1); pn = rank * U + (t >> 1); ch = 0; return; }
        t -= QOWN;
    }
    const int i = t >> 5, q = t & 31;
    pm = 8 * (j & 1) + (q & 7);
    pn = NOWN + 16 * i + 4 * (j >> 1) + (q >> 3);
    ch = 1 + ((pn - NOWN) >> 3);
}

__global__ void __launch_bounds__(512, 2) fused_kernel(Params p) {
    extern __shared__ __attribute__((aligned(16))) unsigned char shm[];
    LAS unsigned char* lds = (LAS unsigned char*)shm;
    char* myws = p.ws[p.rank];
    const int tid = threadIdx.x;
    LAS unsigned* s_words = (LAS unsigned*)(lds + STAGE_BYTES);

    if (tid == 0) {
        unsigned t = __hip_atomic_fetch_add(ctrl(myws, C_TICKET), 1u, __ATOMIC_RELAXED, __HIP_MEMORY_SCOPE_AGENT);
        s_words[0] = t / gridDim.x + 1;
        s_words[1] = 1u;
    }
    __syncthreads();
    const unsigned call = s_words[0];
    const size_t bufoff = WS_BUF + (size_t)(call & 1) * BUF_BYTES;

    if (blockIdx.x < NP) {
        const int pw = blockIdx.x;
        for (int sq = -2; sq < NWCH; ++sq) {
            const int c = sq < 0 ? sq : (sq < 2 * S0 ? sq + 1 : (sq == 2 * S0 ? 0 : sq));
            if (c < 0) {
                const int h = c + 2;
                #pragma unroll 4
                for (int it = 0; it < 2048 / NP / 8; ++it) {
                    const int row = h * 2048 + pw * (2048 / NP) + it * 8 + (tid >> 6), c8 = (tid & 63) * 8;
                    const f32x4 a = *reinterpret_cast<const f32x4*>(p.x + (size_t)row * KS + c8);
                    const f32x4 b = *reinterpret_cast<const f32x4*>(p.x + (size_t)row * KS + c8 + 4);
                    f16x8 v;
                    v[0] = (_Float16)a[0]; v[1] = (_Float16)a[1]; v[2] = (_Float16)a[2]; v[3] = (_Float16)a[3];
                    v[4] = (_Float16)b[0]; v[5] = (_Float16)b[1]; v[6] = (_Float16)b[2]; v[7] = (_Float16)b[3];
                    const size_t off = bufoff + ((size_t)row * GK + p.rank * KS + c8) * 2;
                    #pragma unroll
                    for (int q = 0; q < NR; ++q) st_wt(p.ws[(p.rank + q) & (NR - 1)] + off, v);
                }
            } else {
                const int nb = c == 0 ? NOWN * 8 / NP : 64 / NP;
                for (int b = 0; b < nb; ++b) {
                    const int n0 = c == 0 ? (pw * nb + b) * 32 : (NOWN * 256 + (c - 1) * 2048) + (pw * nb + b) * 32;
                    const int nq = tid & 7, kq = tid >> 3;
                    f32x4 r[8];
                    #pragma unroll
                    for (int j = 0; j < 8; ++j) r[j] = *reinterpret_cast<const f32x4*>(p.w + (size_t)(8 * kq + j) * GN + n0 + 4 * nq);
                    __syncthreads();
                    #pragma unroll
                    for (int i = 0; i < 4; ++i) {
                        f16x8 v;
                        #pragma unroll
                        for (int j = 0; j < 8; ++j) v[j] = (_Float16)r[j][i];
                        *(LAS f16x8*)(lds + (4 * nq + i) * 1040 + kq * 16) = v;
                    }
                    __syncthreads();
                    #pragma unroll
                    for (int j = 0; j < 4; ++j) {
                        const int idx = tid + 512 * j, n = idx >> 6, seg = idx & 63;
                        const f16x8 v = *(const LAS f16x8*)(lds + n * 1040 + seg * 16);
                        const size_t off = bufoff + XH_BYTES + ((size_t)(n0 + n) * GK + p.rank * KS) * 2 + seg * 16;
                        if (c == 0) { st_wt(p.ws[n0 / (U * 256)] + off, v); }
                        else {
                            #pragma unroll
                            for (int q = 0; q < NR; ++q) st_wt(p.ws[(p.rank + q) & (NR - 1)] + off, v);
                        }
                    }
                }
            }
            asm volatile("s_waitcnt vmcnt(0)" ::: "memory");
            __syncthreads();
            if (tid == 0) {
                const int ci = c < 0 ? C_X + c + 2 : C_W + c;
                #pragma unroll
                for (int q = 0; q < NR; ++q)
                    __hip_atomic_fetch_add(ctrl(p.ws[(p.rank + q) & (NR - 1)], ci), 1u, __ATOMIC_RELAXED, __HIP_MEMORY_SCOPE_SYSTEM);
            }
        }
        return;
    }

    unsigned* tmo = ctrl(myws, C_TMO);
    const int xj = (int)((unsigned)__builtin_amdgcn_s_getreg((3 << 11) | 20) & 7u);
    unsigned* qhead = ctrl(myws, C_Q + xj);
    const unsigned need = (unsigned)(NR * NP) * call;
    const int wid = __builtin_amdgcn_readfirstlane(tid >> 6), lane = tid & 63, wr = wid >> 2, wc = wid & 3, fr = lane & 15, fq = lane >> 4;
    constexpr int nt = GK / BK;
    unsigned voff[2];
    #pragma unroll
    for (int i = 0; i < 2; ++i) { int R, C; stage_rc(tid * 16 + i * 8192, R, C); voff[i] = (unsigned)(R * GK + C) * 2u; }
    constexpr size_t kstep = BK * 2, hstep = (size_t)HALF * GK * 2, tstep = 2 * hstep;
    const unsigned ldsw = (unsigned)wid * 1024u;
    const int aoff = lds_byte(wr * 64 + fr, fq * 8), boff = lds_byte(wc * 32 + fr, fq * 8);
    const char* gA = myws + bufoff;
    const char* gB = myws + bufoff + XH_BYTES;
#define SA(b, h) (((b) * 2 + (h)) * HTB)
#define SB(b, h) ((4 + (b) * 2 + (h)) * HTB)
#define STAGE(bo, gbase) do { _Pragma("unroll") for (int _i = 0; _i < 2; ++_i) \
        __builtin_amdgcn_global_load_lds((const unsigned*)((const char*)(gbase) + voff[_i]), (LAS unsigned*)(lds + (bo) + ldsw + _i * 8192), 16, 0, 0); } while (0)
#define LDA(dst, b, h) do { _Pragma("unroll") for (int m = 0; m < 4; ++m) _Pragma("unroll") for (int k = 0; k < 2; ++k) dst[m][k] = *(const LAS f16x8*)(lds + SA(b, h) + aoff + m * 2048 + k * 1024); } while (0)
#define LDB(dst, b, h) do { _Pragma("unroll") for (int n = 0; n < 2; ++n) _Pragma("unroll") for (int k = 0; k < 2; ++k) dst[n][k] = *(const LAS f16x8*)(lds + SB(b, h) + boff + n * 2048 + k * 1024); } while (0)
#define MMA(ai, bj, At, Bt) do { __builtin_amdgcn_s_setprio(1); _Pragma("unroll") for (int m = 0; m < 4; ++m) _Pragma("unroll") for (int n = 0; n < 2; ++n) _Pragma("unroll") for (int k = 0; k < 2; ++k) \
        acc[ai][bj][m][n] = __builtin_amdgcn_mfma_f32_16x16x32_f16(Bt[n][k], At[m][k], acc[ai][bj][m][n], 0, 0, 0); __builtin_amdgcn_s_setprio(0); } while (0)
#define WAIT_V(n) asm volatile("s_waitcnt vmcnt(" #n ")" ::: "memory")
#define WAIT_L(n) asm volatile("s_waitcnt lgkmcnt(" #n ")" ::: "memory")
#define BAR __builtin_amdgcn_s_barrier()
#define SCHED __builtin_amdgcn_sched_barrier(0)
#define READY(PM, CH, ACQ) do { if (tid < 64) { \
        unsigned* f0 = ctrl(myws, C_X + ((PM) >> 3)); unsigned* f1 = ctrl(myws, C_W + (CH)); \
        const unsigned long long t0 = __builtin_amdgcn_s_memrealtime(); \
        while ((unsigned)__builtin_amdgcn_readfirstlane(__hip_atomic_load(f0, __ATOMIC_RELAXED, __HIP_MEMORY_SCOPE_SYSTEM)) < need || \
               (unsigned)__builtin_amdgcn_readfirstlane(__hip_atomic_load(f1, __ATOMIC_RELAXED, __HIP_MEMORY_SCOPE_SYSTEM)) < need) { \
            if ((unsigned)__builtin_amdgcn_readfirstlane(__hip_atomic_load(tmo, __ATOMIC_RELAXED, __HIP_MEMORY_SCOPE_AGENT)) == call || \
                __builtin_amdgcn_s_memrealtime() - t0 > DEADLINE_TICKS) { \
                if (tid == 0) { __hip_atomic_store(tmo, call, __ATOMIC_RELAXED, __HIP_MEMORY_SCOPE_AGENT); s_words[1] = 0u; } \
                break; } \
            __builtin_amdgcn_s_sleep(2); } \
        if (ACQ) __builtin_amdgcn_fence(__ATOMIC_ACQUIRE, ""); \
        asm volatile("s_waitcnt vmcnt(0) lgkmcnt(0)" ::: "memory"); } \
        asm volatile("" ::: "memory"); __builtin_amdgcn_s_barrier(); asm volatile("" ::: "memory"); } while (0)

    if (tid == 0) s_words[2] = __hip_atomic_fetch_add(qhead, 1u, __ATOMIC_RELAXED, __HIP_MEMORY_SCOPE_AGENT);
    __syncthreads();
    int cur = (int)s_words[2];
    if (cur < QLEN) {
        int pm, pn, ch, ui = 0;
        tile_of(cur, xj, p.rank, pm, pn, ch);
        f32x4 acc[2][2][4][2];
        #pragma unroll
        for (int a = 0; a < 2; ++a)
            #pragma unroll
            for (int b = 0; b < 2; ++b)
                #pragma unroll
                for (int m = 0; m < 4; ++m)
                    #pragma unroll
                    for (int n = 0; n < 2; ++n) acc[a][b][m][n] = (f32x4){0.f, 0.f, 0.f, 0.f};
        f16x8 At[4][2], B0[2][2], B1[2][2];
        const char* cA = gA + (size_t)pm * tstep; const char* cB = gB + (size_t)pn * tstep;
        READY(pm, ch, true);
        STAGE(SB(0, 0), cB); STAGE(SA(0, 0), cA); STAGE(SB(0, 1), cB + hstep); STAGE(SA(0, 1), cA + hstep);
        if (wr == 1) BAR;
        WAIT_V(4); BAR;
        STAGE(SB(1, 0), cB + kstep); STAGE(SA(1, 0), cA + kstep); STAGE(SB(1, 1), cB + hstep + kstep);
        WAIT_V(6); BAR;
        for (;;) {
            bool has_next = false;
            int npm = pm, npn = pn, nch = ch;
            const char* nA = cA; const char* nB = cB;
            for (int t = 0; t < nt; t += 2) {
                const bool last = (t == nt - 2);
                if (t == 8 && tid == 0) s_words[2 + ((ui + 1) & 1)] = __hip_atomic_fetch_add(qhead, 1u, __ATOMIC_RELAXED, __HIP_MEMORY_SCOPE_AGENT);
                if (last) {
                    const int nx = (int)s_words[2 + ((ui + 1) & 1)];
                    has_next = nx < QLEN;
                    if (has_next) {
                        tile_of(nx, xj, p.rank, npm, npn, nch);
                        nA = gA + (size_t)npm * tstep; nB = gB + (size_t)npn * tstep;
                        READY(npm, nch, false);
                    }
                }
                const char* a1 = cA + (size_t)(t + 1) * kstep;
                const char* a2 = last ? nA : cA + (size_t)(t + 2) * kstep; const char* b2 = last ? nB : cB + (size_t)(t + 2) * kstep;
                const char* a3 = a2 + kstep; const char* b3 = b2 + kstep;
                LDB(B0, 0, 0); SCHED; LDA(At, 0, 0); STAGE(SA(1, 1), a1 + hstep);
                WAIT_L(8); BAR; WAIT_L(0); MMA(0, 0, At, B0); BAR; SCHED;
                LDB(B1, 0, 1); STAGE(SB(0, 0), b2);
                BAR; WAIT_L(0); MMA(0, 1, At, B1); BAR;
                LDA(At, 0, 1); STAGE(SA(0, 0), a2);
                BAR; WAIT_L(0); MMA(1, 0, At, B0); BAR; SCHED;
                STAGE(SB(0, 1), b2 + hstep);
                WAIT_V(6); BAR; MMA(1, 1, At, B1); BAR;
                LDB(B0, 1, 0); SCHED; LDA(At, 1, 0); STAGE(SA(0, 1), a2 + hstep);
                WAIT_L(8); BAR; WAIT_L(0); MMA(0, 0, At, B0); BAR; SCHED;
                LDB(B1, 1, 1); STAGE(SB(1, 0), b3);
                BAR; WAIT_L(0); MMA(0, 1, At, B1); BAR;
                LDA(At, 1, 1); STAGE(SA(1, 0), a3);
                BAR; WAIT_L(0); MMA(1, 0, At, B0); BAR; SCHED;
                STAGE(SB(1, 1), b3 + hstep);
                WAIT_V(6); BAR; MMA(1, 1, At, B1); BAR;
            }
            {
                const bool ok = s_words[1] != 0u;
                const float nanv = __builtin_nanf("");
                const size_t c0 = (size_t)(pm * BM + wr * 64 + fr) * GN + pn * BM + wc * 32 + 4 * fq;
                float* C0 = p.out[p.rank] + c0;
                #pragma unroll
                for (int ai = 0; ai < 2; ++ai)
                    #pragma unroll
                    for (int m = 0; m < 4; ++m) {
                        float* rowp = C0 + (size_t)(ai * HALF + m * 16) * GN;
                        #pragma unroll
                        for (int bj = 0; bj < 2; ++bj)
                            #pragma unroll
                            for (int n = 0; n < 2; ++n) {
                                f32x4 v = acc[ai][bj][m][n];
                                #pragma unroll
                                for (int e = 0; e < 4; ++e) v[e] = ok ? fmaxf(v[e], 0.f) : nanv;

                                *reinterpret_cast<f32x4*>(rowp + bj * HALF + n * 16) = v;
                                if (ch == 0) {
                                    const size_t o = c0 + (size_t)(ai * HALF + m * 16) * GN + bj * HALF + n * 16;
                                    #pragma unroll
                                    for (int q = 1; q < NR; ++q) st_wt32(p.out[(p.rank + q) & (NR - 1)] + o, v);
                                }
                                acc[ai][bj][m][n] = (f32x4){0.f, 0.f, 0.f, 0.f};
                            }
                    }
                if (ch == 0) {
                    WAIT_V(0);
                    if (lane == 0) {
                        #pragma unroll
                        for (int q = 1; q < NR; ++q)
                            __hip_atomic_fetch_add(ctrl(p.ws[(p.rank + q) & (NR - 1)], C_DONE + p.rank), 1u, __ATOMIC_RELAXED, __HIP_MEMORY_SCOPE_SYSTEM);
                    }
                }
            }
            if (!has_next) break;
            pm = npm; pn = npn; ch = nch; cA = nA; cB = nB; ++ui;
        }
        WAIT_V(0);
        if (wr == 0) BAR;
        BAR;
    }
    if (tid == 0) {
        const unsigned e = __hip_atomic_fetch_add(ctrl(myws, C_EXIT), 1u, __ATOMIC_ACQ_REL, __HIP_MEMORY_SCOPE_AGENT);
        if (e % NGW == NGW - 1) {
            #pragma unroll
            for (int j = 0; j < 8; ++j) __hip_atomic_store(ctrl(myws, C_Q + j), 0u, __ATOMIC_RELAXED, __HIP_MEMORY_SCOPE_AGENT);
            const unsigned dneed = (unsigned)(16 * U * 8) * call;
            const unsigned long long t0 = __builtin_amdgcn_s_memrealtime();
            bool ok = true;
            for (int q = 1; q < NR && ok; ++q) {
                unsigned* f = ctrl(myws, C_DONE + ((p.rank + q) & (NR - 1)));
                while (__hip_atomic_load(f, __ATOMIC_RELAXED, __HIP_MEMORY_SCOPE_SYSTEM) < dneed) {
                    if (__hip_atomic_load(tmo, __ATOMIC_RELAXED, __HIP_MEMORY_SCOPE_AGENT) == call ||
                        __builtin_amdgcn_s_memrealtime() - t0 > DEADLINE_TICKS) {
                        __hip_atomic_store(tmo, call, __ATOMIC_RELAXED, __HIP_MEMORY_SCOPE_AGENT);
                        ok = false; break;
                    }
                    __builtin_amdgcn_s_sleep(2);
                }
            }
            __builtin_amdgcn_fence(__ATOMIC_ACQUIRE, "");
            if (!ok || __hip_atomic_load(tmo, __ATOMIC_RELAXED, __HIP_MEMORY_SCOPE_AGENT) == call)
                p.out[p.rank][(size_t)p.rank * U * 256] = __builtin_nanf("");
        }
    }
}

}

extern "C" void kernel_launch(int rank, int nranks, ncclComm_t comm,
                              hipStream_t stream,
                              void* const* d_in, const int* in_sizes, int n_in,
                              void* d_out, int out_size,
                              const int* in_dtypes, int out_dtype,
                              void* const* d_out_all,
                              void* const* d_ws_all, size_t ws_bytes) {
    (void)comm; (void)in_sizes; (void)n_in; (void)out_size; (void)in_dtypes; (void)out_dtype; (void)d_out;
    if (nranks != NR || ws_bytes < WS_END) {
        fprintf(stderr, "kernel_launch: unsupported configuration (nranks=%d ws=%zu)\n", nranks, ws_bytes);
        return;
    }
    Params p;
    p.x = static_cast<const float*>(d_in[0]);
    p.w = static_cast<const float*>(d_in[1]);
    for (int i = 0; i < NR; ++i) { p.ws[i] = static_cast<char*>(d_ws_all[i]); p.out[i] = static_cast<float*>(d_out_all[i]); }
    p.rank = rank;
    if (hipFuncSetAttribute(reinterpret_cast<const void*>(fused_kernel),
                            hipFuncAttributeMaxDynamicSharedMemorySize, LDS_BYTES) != hipSuccess) {
        fprintf(stderr, "kernel_launch: hipFuncSetAttribute failed\n");
        return;
    }
    fused_kernel<<<NP + NGW, 512, LDS_BYTES, stream>>>(p);
}
```

```cpp
#include <hip/hip_runtime.h>
#include <rccl/rccl.h>
#include <cstdio>
#include <cstdint>

namespace {

constexpr int NR = 8, GM = 4096, GK = 4096, GN = 32768, KS = GK / NR;

constexpr int BM = 256, BK = 64, HALF = 128, HTB = HALF * BK * 2, STAGE_BYTES = 8 * HTB;
constexpr int LDS_BYTES = STAGE_BYTES + 64;
constexpr int NP  = 16;
constexpr int NGW = 256;
constexpr int U    = 2;
constexpr int NOWN = 8 * U;
constexpr int NWCH = 1 + (128 - NOWN) / 8;
constexpr int S0 = 5;
constexpr int QOWN = 2 * U;
constexpr int QLEN = QOWN + 2 * (128 - NOWN);

constexpr size_t XH_BYTES = (size_t)GM * GK * 2;
constexpr size_t WH_BYTES = (size_t)GN * GK * 2;
constexpr size_t BUF_BYTES = XH_BYTES + WH_BYTES;
constexpr size_t WS_BUF = 1u << 20;
constexpr size_t WS_END = WS_BUF + 2 * BUF_BYTES;

constexpr int LINE = 64;
constexpr int C_TICKET = 0, C_TMO = 1, C_EXIT = 2;
constexpr int C_Q = 8;
constexpr int C_X = 16;
constexpr int C_W = 24;
constexpr int C_DONE = 48;

constexpr unsigned long long DEADLINE_TICKS = 100000000ull;

typedef float f32x4 __attribute__((ext_vector_type(4)));
typedef _Float16 f16x8 __attribute__((ext_vector_type(8)));
#define LAS __attribute__((address_space(3)))

struct Params {
    const float* x;
    const float* w;
    float* out[NR];
    char*  ws[NR];
    int rank;
};

__device__ __forceinline__ unsigned* ctrl(char* ws, int idx) { return reinterpret_cast<unsigned*>(ws) + idx * LINE; }

__device__ __forceinline__ void st_wt(char* ptr, f16x8 v) {
    asm volatile("global_store_dwordx4 %0, %1, off sc0 sc1" :: "v"(ptr), "v"(v) : "memory");
}
__device__ __forceinline__ void st_wt32(float* ptr, f32x4 v) {
    asm volatile("global_store_dwordx4 %0, %1, off sc0 sc1" :: "v"(ptr), "v"(v) : "memory");
}
__device__ __forceinline__ int lds_byte(int r, int c) { const int st = (r >> 4) * 2 + (c >> 5), rr = r & 15, cc = c & 31, ob = rr * 64 + cc * 2; return st * 1024 + (ob ^ (((ob >> 9) & 1) << 5)); }
__device__ __forceinline__ void stage_rc(int b, int& R, int& C) { const int st = b / 1024, sb = b % 1024, swz = sb ^ (((sb >> 9) & 1) << 5); R = (st >> 1) * 16 + swz / 64; C = (st & 1) * 32 + (swz % 64) / 2; }

__device__ __forceinline__ void tile_of(int t, int j, int rank, int& pm, int& pn, int& ch) {
    if (t >= 32 * S0) {
        if (t < 32 * S0 + QOWN) { t -= 32 * S0; pm = 2 * j + (t & 1); pn = rank * U + (t >> 1); ch = 0; return; }
        t -= QOWN;
    }
    const int i = t >> 5, q = t & 31;
    pm = 8 * (j & 1) + (q & 7);
    pn = NOWN + 16 * i + 4 * (j >> 1) + (q >> 3);
    ch = 1 + ((pn - NOWN) >> 3);
}

__global__ void __launch_bounds__(512, 2) fused_kernel(Params p) {
    extern __shared__ __attribute__((aligned(16))) unsigned char shm[];
    LAS unsigned char* lds = (LAS unsigned char*)shm;
    char* myws = p.ws[p.rank];
    const int tid = threadIdx.x;
    LAS unsigned* s_words = (LAS unsigned*)(lds + STAGE_BYTES);

    if (tid == 0) {
        unsigned t = __hip_atomic_fetch_add(ctrl(myws, C_TICKET), 1u, __ATOMIC_RELAXED, __HIP_MEMORY_SCOPE_AGENT);
        s_words[0] = t / gridDim.x + 1;
        s_words[1] = 1u;
    }
    __syncthreads();
    const unsigned call = s_words[0];
    const size_t bufoff = WS_BUF + (size_t)(call & 1) * BUF_BYTES;

    if (blockIdx.x < NP) {
        const int pw = blockIdx.x;
        for (int sq = -2; sq < NWCH; ++sq) {
            const int c = sq < 0 ? sq : (sq < 2 * S0 ? sq + 1 : (sq == 2 * S0 ? 0 : sq));
            if (c < 0) {
                const int h = c + 2;
                #pragma unroll 4
                for (int it = 0; it < 2048 / NP / 8; ++it) {
                    const int row = h * 2048 + pw * (2048 / NP) + it * 8 + (tid >> 6), c8 = (tid & 63) * 8;
                    const f32x4 a = *reinterpret_cast<const f32x4*>(p.x + (size_t)row * KS + c8);
                    const f32x4 b = *reinterpret_cast<const f32x4*>(p.x + (size_t)row * KS + c8 + 4);
                    f16x8 v;
                    v[0] = (_Float16)a[0]; v[1] = (_Float16)a[1]; v[2] = (_Float16)a[2]; v[3] = (_Float16)a[3];
                    v[4] = (_Float16)b[0]; v[5] = (_Float16)b[1]; v[6] = (_Float16)b[2]; v[7] = (_Float16)b[3];
                    const size_t off = bufoff + ((size_t)row * GK + p.rank * KS + c8) * 2;
                    #pragma unroll
                    for (int q = 0; q < NR; ++q) st_wt(p.ws[(p.rank + q) & (NR - 1)] + off, v);
                }
            } else {
                const int nb = c == 0 ? NOWN * 8 / NP : 64 / NP;
                for (int b = 0; b < nb; ++b) {
                    const int n0 = c == 0 ? (pw * nb + b) * 32 : (NOWN * 256 + (c - 1) * 2048) + (pw * nb + b) * 32;
                    const int nq = tid & 7, kq = tid >> 3;
                    f32x4 r[8];
                    #pragma unroll
                    for (int j = 0; j < 8; ++j) r[j] = *reinterpret_cast<const f32x4*>(p.w + (size_t)(8 * kq + j) * GN + n0 + 4 * nq);
                    __syncthreads();
                    #pragma unroll
                    for (int i = 0; i < 4; ++i) {
                        f16x8 v;
                        #pragma unroll
                        for (int j = 0; j < 8; ++j) v[j] = (_Float16)r[j][i];
                        *(LAS f16x8*)(lds + (4 * nq + i) * 1040 + kq * 16) = v;
                    }
                    __syncthreads();
                    #pragma unroll
                    for (int j = 0; j < 4; ++j) {
                        const int idx = tid + 512 * j, n = idx >> 6, seg = idx & 63;
                        const f16x8 v = *(const LAS f16x8*)(lds + n * 1040 + seg * 16);
                        const size_t off = bufoff + XH_BYTES + ((size_t)(n0 + n) * GK + p.rank * KS) * 2 + seg * 16;
                        if (c == 0) { st_wt(p.ws[n0 / (U * 256)] + off, v); }
                        else {
                            #pragma unroll
                            for (int q = 0; q < NR; ++q) st_wt(p.ws[(p.rank + q) & (NR - 1)] + off, v);
                        }
                    }
                }
            }
            asm volatile("s_waitcnt vmcnt(0)" ::: "memory");
            __syncthreads();
            if (tid == 0) {
                const int ci = c < 0 ? C_X + c + 2 : C_W + c;
                #pragma unroll
                for (int q = 0; q < NR; ++q)
                    __hip_atomic_fetch_add(ctrl(p.ws[(p.rank + q) & (NR - 1)], ci), 1u, __ATOMIC_RELAXED, __HIP_MEMORY_SCOPE_SYSTEM);
            }
        }
        return;
    }

    unsigned* tmo = ctrl(myws, C_TMO);
    const int xj = (int)((unsigned)__builtin_amdgcn_s_getreg((3 << 11) | 20) & 7u);
    const unsigned need = (unsigned)(NR * NP) * call;
    const int wid = __builtin_amdgcn_readfirstlane(tid >> 6), lane = tid & 63, wr = wid >> 2, wc = wid & 3, fr = lane & 15, fq = lane >> 4;
    constexpr int nt = GK / BK;
    unsigned voff[2];
    #pragma unroll
    for (int i = 0; i < 2; ++i) { int R, C; stage_rc(tid * 16 + i * 8192, R, C); voff[i] = (unsigned)(R * GK + C) * 2u; }
    constexpr size_t kstep = BK * 2, hstep = (size_t)HALF * GK * 2, tstep = 2 * hstep;
    const unsigned ldsw = (unsigned)wid * 1024u;
    const int aoff = lds_byte(wr * 64 + fr, fq * 8), boff = lds_byte(wc * 32 + fr, fq * 8);
    const char* gA = myws + bufoff;
    const char* gB = myws + bufoff + XH_BYTES;
#define SA(b, h) (((b) * 2 + (h)) * HTB)
#define SB(b, h) ((4 + (b) * 2 + (h)) * HTB)
#define STAGE(bo, gbase) do { _Pragma("unroll") for (int _i = 0; _i < 2; ++_i) \
        __builtin_amdgcn_global_load_lds((const unsigned*)((const char*)(gbase) + voff[_i]), (LAS unsigned*)(lds + (bo) + ldsw + _i * 8192), 16, 0, 0); } while (0)
#define LDA(dst, b, h) do { _Pragma("unroll") for (int m = 0; m < 4; ++m) _Pragma("unroll") for (int k = 0; k < 2; ++k) dst[m][k] = *(const LAS f16x8*)(lds + SA(b, h) + aoff + m * 2048 + k * 1024); } while (0)
#define LDB(dst, b, h) do { _Pragma("unroll") for (int n = 0; n < 2; ++n) _Pragma("unroll") for (int k = 0; k < 2; ++k) dst[n][k] = *(const LAS f16x8*)(lds + SB(b, h) + boff + n * 2048 + k * 1024); } while (0)
#define MMA(ai, bj, At, Bt) do { __builtin_amdgcn_s_setprio(1); _Pragma("unroll") for (int m = 0; m < 4; ++m) _Pragma("unroll") for (int n = 0; n < 2; ++n) _Pragma("unroll") for (int k = 0; k < 2; ++k) \
        acc[ai][bj][m][n] = __builtin_amdgcn_mfma_f32_16x16x32_f16(Bt[n][k], At[m][k], acc[ai][bj][m][n], 0, 0, 0); __builtin_amdgcn_s_setprio(0); } while (0)
#define WAIT_V(n) asm volatile("s_waitcnt vmcnt(" #n ")" ::: "memory")
#define WAIT_L(n) asm volatile("s_waitcnt lgkmcnt(" #n ")" ::: "memory")
#define BAR __builtin_amdgcn_s_barrier()
#define SCHED __builtin_amdgcn_sched_barrier(0)
#define READY(PM, CH, ACQ) do { if (tid < 64) { \
        unsigned* f0 = ctrl(myws, C_X + ((PM) >> 3)); unsigned* f1 = ctrl(myws, C_W + (CH)); \
        const unsigned long long t0 = __builtin_amdgcn_s_memrealtime(); \
        while ((unsigned)__builtin_amdgcn_readfirstlane(__hip_atomic_load(f0, __ATOMIC_RELAXED, __HIP_MEMORY_SCOPE_SYSTEM)) < need || \
               (unsigned)__builtin_amdgcn_readfirstlane(__hip_atomic_load(f1, __ATOMIC_RELAXED, __HIP_MEMORY_SCOPE_SYSTEM)) < need) { \
            if ((unsigned)__builtin_amdgcn_readfirstlane(__hip_atomic_load(tmo, __ATOMIC_RELAXED, __HIP_MEMORY_SCOPE_AGENT)) == call || \
                __builtin_amdgcn_s_memrealtime() - t0 > DEADLINE_TICKS) { \
                if (tid == 0) { __hip_atomic_store(tmo, call, __ATOMIC_RELAXED, __HIP_MEMORY_SCOPE_AGENT); s_words[1] = 0u; } \
                break; } \
            __builtin_amdgcn_s_sleep(2); } \
        if (ACQ) __builtin_amdgcn_fence(__ATOMIC_ACQUIRE, ""); \
        asm volatile("s_waitcnt vmcnt(0) lgkmcnt(0)" ::: "memory"); } \
        asm volatile("" ::: "memory"); __builtin_amdgcn_s_barrier(); asm volatile("" ::: "memory"); } while (0)

#define FETCH(DST) do { unsigned r_ = ~0u; \
        for (int k_ = 0; k_ < 8; ++k_) { const int q_ = (xj + k_) & 7; \
            const unsigned t_ = __hip_atomic_fetch_add(ctrl(myws, C_Q + q_), 1u, __ATOMIC_RELAXED, __HIP_MEMORY_SCOPE_AGENT); \
            if (t_ < (unsigned)QLEN) { r_ = ((unsigned)q_ << 16) | t_; break; } } \
        DST = r_; } while (0)
    if (tid == 0) FETCH(s_words[2]);
    __syncthreads();
    const unsigned cur = s_words[2];
    if (cur != ~0u) {
        int pm, pn, ch, ui = 0;
        tile_of((int)(cur & 0xffffu), (int)(cur >> 16), p.rank, pm, pn, ch);
        f32x4 acc[2][2][4][2];
        #pragma unroll
        for (int a = 0; a < 2; ++a)
            #pragma unroll
            for (int b = 0; b < 2; ++b)
                #pragma unroll
                for (int m = 0; m < 4; ++m)
                    #pragma unroll
                    for (int n = 0; n < 2; ++n) acc[a][b][m][n] = (f32x4){0.f, 0.f, 0.f, 0.f};
        f16x8 At[4][2], B0[2][2], B1[2][2];
        const char* cA = gA + (size_t)pm * tstep; const char* cB = gB + (size_t)pn * tstep;
        READY(pm, ch, true);
        STAGE(SB(0, 0), cB); STAGE(SA(0, 0), cA); STAGE(SB(0, 1), cB + hstep); STAGE(SA(0, 1), cA + hstep);
        if (wr == 1) BAR;
        WAIT_V(4); BAR;
        STAGE(SB(1, 0), cB + kstep); STAGE(SA(1, 0), cA + kstep); STAGE(SB(1, 1), cB + hstep + kstep);
        WAIT_V(6); BAR;
        for (;;) {
            bool has_next = false;
            int npm = pm, npn = pn, nch = ch;
            const char* nA = cA; const char* nB = cB;
            for (int t = 0; t < nt; t += 2) {
                const bool last = (t == nt - 2);
                if (t == 8 && tid == 0) FETCH(s_words[2 + ((ui + 1) & 1)]);
                if (last) {
                    const unsigned nx = s_words[2 + ((ui + 1) & 1)];
                    has_next = nx != ~0u;
                    if (has_next) {
                        tile_of((int)(nx & 0xffffu), (int)(nx >> 16), p.rank, npm, npn, nch);
                        nA = gA + (size_t)npm * tstep; nB = gB + (size_t)npn * tstep;
                        READY(npm, nch, false);
                    }
                }
                const char* a1 = cA + (size_t)(t + 1) * kstep;
                const char* a2 = last ? nA : cA + (size_t)(t + 2) * kstep; const char* b2 = last ? nB : cB + (size_t)(t + 2) * kstep;
                const char* a3 = a2 + kstep; const char* b3 = b2 + kstep;
                LDB(B0, 0, 0); SCHED; LDA(At, 0, 0); STAGE(SA(1, 1), a1 + hstep);
                WAIT_L(8); BAR; WAIT_L(0); MMA(0, 0, At, B0); BAR; SCHED;
                LDB(B1, 0, 1); STAGE(SB(0, 0), b2);
                BAR; WAIT_L(0); MMA(0, 1, At, B1); BAR;
                LDA(At, 0, 1); STAGE(SA(0, 0), a2);
                BAR; WAIT_L(0); MMA(1, 0, At, B0); BAR; SCHED;
                STAGE(SB(0, 1), b2 + hstep);
                WAIT_V(6); BAR; MMA(1, 1, At, B1); BAR;
                LDB(B0, 1, 0); SCHED; LDA(At, 1, 0); STAGE(SA(0, 1), a2 + hstep);
                WAIT_L(8); BAR; WAIT_L(0); MMA(0, 0, At, B0); BAR; SCHED;
                LDB(B1, 1, 1); STAGE(SB(1, 0), b3);
                BAR; WAIT_L(0); MMA(0, 1, At, B1); BAR;
                LDA(At, 1, 1); STAGE(SA(1, 0), a3);
                BAR; WAIT_L(0); MMA(1, 0, At, B0); BAR; SCHED;
                STAGE(SB(1, 1), b3 + hstep);
                WAIT_V(6); BAR; MMA(1, 1, At, B1); BAR;
            }
            {
                const bool ok = s_words[1] != 0u;
                const float nanv = __builtin_nanf("");
                const size_t c0 = (size_t)(pm * BM + wr * 64 + fr) * GN + pn * BM + wc * 32 + 4 * fq;
                float* C0 = p.out[p.rank] + c0;
                #pragma unroll
                for (int ai = 0; ai < 2; ++ai)
                    #pragma unroll
                    for (int m = 0; m < 4; ++m) {
                        float* rowp = C0 + (size_t)(ai * HALF + m * 16) * GN;
                        #pragma unroll
                        for (int bj = 0; bj < 2; ++bj)
                            #pragma unroll
                            for (int n = 0; n < 2; ++n) {
                                f32x4 v = acc[ai][bj][m][n];
                                #pragma unroll
                                for (int e = 0; e < 4; ++e) v[e] = ok ? fmaxf(v[e], 0.f) : nanv;

                                *reinterpret_cast<f32x4*>(rowp + bj * HALF + n * 16) = v;
                                if (ch == 0) {
                                    const size_t o = c0 + (size_t)(ai * HALF + m * 16) * GN + bj * HALF + n * 16;
                                    #pragma unroll
                                    for (int q = 1; q < NR; ++q) st_wt32(p.out[(p.rank + q) & (NR - 1)] + o, v);
                                }
                                acc[ai][bj][m][n] = (f32x4){0.f, 0.f, 0.f, 0.f};
                            }
                    }
                if (ch == 0) {
                    WAIT_V(0);
                    if (lane == 0) {
                        #pragma unroll
                        for (int q = 1; q < NR; ++q)
                            __hip_atomic_fetch_add(ctrl(p.ws[(p.rank + q) & (NR - 1)], C_DONE + p.rank), 1u, __ATOMIC_RELAXED, __HIP_MEMORY_SCOPE_SYSTEM);
                    }
                }
            }
            if (!has_next) break;
            pm = npm; pn = npn; ch = nch; cA = nA; cB = nB; ++ui;
        }
        WAIT_V(0);
        if (wr == 0) BAR;
        BAR;
    }
    if (tid == 0) {
        const unsigned e = __hip_atomic_fetch_add(ctrl(myws, C_EXIT), 1u, __ATOMIC_ACQ_REL, __HIP_MEMORY_SCOPE_AGENT);
        if (e % NGW == NGW - 1) {
            #pragma unroll
            for (int j = 0; j < 8; ++j) __hip_atomic_store(ctrl(myws, C_Q + j), 0u, __ATOMIC_RELAXED, __HIP_MEMORY_SCOPE_AGENT);
            const unsigned dneed = (unsigned)(16 * U * 8) * call;
            const unsigned long long t0 = __builtin_amdgcn_s_memrealtime();
            bool ok = true;
            for (int q = 1; q < NR && ok; ++q) {
                unsigned* f = ctrl(myws, C_DONE + ((p.rank + q) & (NR - 1)));
                while (__hip_atomic_load(f, __ATOMIC_RELAXED, __HIP_MEMORY_SCOPE_SYSTEM) < dneed) {
                    if (__hip_atomic_load(tmo, __ATOMIC_RELAXED, __HIP_MEMORY_SCOPE_AGENT) == call ||
                        __builtin_amdgcn_s_memrealtime() - t0 > DEADLINE_TICKS) {
                        __hip_atomic_store(tmo, call, __ATOMIC_RELAXED, __HIP_MEMORY_SCOPE_AGENT);
                        ok = false; break;
                    }
                    __builtin_amdgcn_s_sleep(2);
                }
            }
            __builtin_amdgcn_fence(__ATOMIC_ACQUIRE, "");
            if (!ok || __hip_atomic_load(tmo, __ATOMIC_RELAXED, __HIP_MEMORY_SCOPE_AGENT) == call)
                p.out[p.rank][(size_t)p.rank * U * 256] = __builtin_nanf("");
        }
    }
}

}

extern "C" void kernel_launch(int rank, int nranks, ncclComm_t comm,
                              hipStream_t stream,
                              void* const* d_in, const int* in_sizes, int n_in,
                              void* d_out, int out_size,
                              const int* in_dtypes, int out_dtype,
                              void* const* d_out_all,
                              void* const* d_ws_all, size_t ws_bytes) {
    (void)comm; (void)in_sizes; (void)n_in; (void)out_size; (void)in_dtypes; (void)out_dtype; (void)d_out;
    if (nranks != NR || ws_bytes < WS_END) {
        fprintf(stderr, "kernel_launch: unsupported configuration (nranks=%d ws=%zu)\n", nranks, ws_bytes);
        return;
    }
    Params p;
    p.x = static_cast<const float*>(d_in[0]);
    p.w = static_cast<const float*>(d_in[1]);
    for (int i = 0; i < NR; ++i) { p.ws[i] = static_cast<char*>(d_ws_all[i]); p.out[i] = static_cast<float*>(d_out_all[i]); }
    p.rank = rank;
    if (hipFuncSetAttribute(reinterpret_cast<const void*>(fused_kernel),
                            hipFuncAttributeMaxDynamicSharedMemorySize, LDS_BYTES) != hipSuccess) {
        fprintf(stderr, "kernel_launch: hipFuncSetAttribute failed\n");
        return;
    }
    fused_kernel<<<NP + NGW, 512, LDS_BYTES, stream>>>(p);
}
```
